```python
import numpy as np

kernel_name = "gpu_gemm_ar_m4096_k4096_n4096_bf16_reluq8"


ABSMAX_RTOL = 0.02

COMPUTE_DTYPE = "bfloat16"
EPILOGUE = "relu_quant_int8"
IN_QUANT = None
OUT_QUANT = "int8"
IN_DTYPES = {"x": "bfloat16", "w_mat": "bfloat16"}
WORLD_SIZE = 8

def _to_bf16(x):
    u = np.ascontiguousarray(x, dtype=np.float32).view(np.uint32)
    return ((u + 0x7FFF + ((u >> 16) & 1)) & 0xFFFF0000).view(np.float32)

def setup_inputs():

    x = _to_bf16(np.random.randn(4096, 4096).astype(np.float32))
    w_mat = _to_bf16((np.random.randn(4096, 4096) / np.sqrt(4096)).astype(np.float32))
    return {"x": x, "w_mat": w_mat}

def _quant_dequant(y, amax):
    scale = amax / 127.0
    q = np.clip(np.round(y / scale), -127.0, 127.0)
    return (q * scale).astype(np.float32)
def _epilogue(y):
    y = np.maximum(y, 0.0)
    return _quant_dequant(y, np.abs(y).max())

def reference(x, w_mat):
    y = (x @ w_mat).astype(np.float32)
    return _epilogue(y)
```

```cpp
#include <hip/hip_runtime.h>
#include <rccl/rccl.h>
#include <cstdio>
#include <cstdint>

#define GLDS_POL "sc1"
#define POLL_POL "sc0 sc1"
#define FLAG_STORE 1
#define COPY_D 2
#define COPY_MAP 1
#define GAP_SLOW 100
#define GAP_FAST 30
#define FAST_LEFT 3
#define PPITCH 128
#define CPRIO 3
#define P2_D 2

namespace {

typedef short v4s __attribute__((ext_vector_type(4)));
typedef short v8s __attribute__((ext_vector_type(8)));
typedef __bf16 v8bf __attribute__((ext_vector_type(8)));
typedef float v4f __attribute__((ext_vector_type(4)));
typedef unsigned v4u __attribute__((ext_vector_type(4)));
typedef unsigned v8u __attribute__((ext_vector_type(8)));
typedef unsigned v16u __attribute__((ext_vector_type(16)));
typedef unsigned long long u64;
#define LAS __attribute__((address_space(3)))

constexpr int NR = 8;
constexpr int NCOL = 4096;
constexpr int KC = 512;
constexpr int NSUB = 16;
constexpr int NUNIT = NR * NSUB;
constexpr int NWG = 256;
constexpr int NL = 7;
constexpr int W_SCHED = 4 + NL, W_COPY = 5 + NL, W_FLAG = 6 + NL, W_FETCH = 7 + NL, W_POLL = 8 + NL, NWAVE = 9 + NL;
constexpr unsigned UNIT_X = 2048u * 32u * 2u;
constexpr unsigned UNIT_W = 32u * 1024u * 2u;
constexpr unsigned UNIT = UNIT_X + UNIT_W;
constexpr unsigned OFF_CTR = 0;
constexpr unsigned OFF_TMO = 32768;
constexpr unsigned OFF_PROGX = 262144;
constexpr unsigned OFF_PROGW = 270336;
constexpr unsigned OFF_WGMAX = 40960;
constexpr unsigned OFF_RMAX = 45056;
constexpr unsigned OFF_DONE = 49152;
constexpr unsigned OFF_DIAG = 65536;
constexpr unsigned OFF_STAGE = 1u << 20;
constexpr unsigned WS_NEED = OFF_STAGE + NR * NSUB * UNIT;
constexpr unsigned SLOT = 12288;
#define NSLOT_ 10
constexpr int NSLOT = NSLOT_;
constexpr unsigned LDS_AUX = NSLOT * SLOT;
constexpr unsigned LDS_FLG = LDS_AUX + 256;
constexpr unsigned LDS_CPY = LDS_FLG + 256;
constexpr int CG = 5;
constexpr unsigned LDS_BYTES = LDS_CPY + CG * 6144;
constexpr int A_ASG = 0, A_CONS = 4, A_MAX = 8, A_CNT = 9, A_GM = 12, A_DEAD = 13, A_GO = 14, A_SENT = 15, A_DESC = 16, A_FULL = 32, A_FETCHED = 48, A_TAKEN = 49, A_PRG = 56;
constexpr u64 DEADLINE = 100000000ull;

struct Params {
    char* ws[NR];
    char* out[NR];
    const char* x;
    const char* w;
    int rank;
};

#define SYS __HIP_MEMORY_SCOPE_SYSTEM
#define AGT __HIP_MEMORY_SCOPE_AGENT
#define WGS __HIP_MEMORY_SCOPE_WORKGROUP
#define RLX __ATOMIC_RELAXED
#define VMCNT(n) asm volatile("s_waitcnt vmcnt(" #n ")" ::: "memory")
#define LGKM0 asm volatile("s_waitcnt lgkmcnt(0)" ::: "memory")
#define CFENCE asm volatile("" ::: "memory")

#define STAMP(slot) do { } while (0)

__device__ __forceinline__ auto mk_rsrc(const void* base, unsigned bytes) {
    return __builtin_amdgcn_make_buffer_rsrc(const_cast<void*>(base), 0, bytes, 0x00020000);
}

__device__ __forceinline__ void glds4(const char* gbase, unsigned voff, unsigned lds) {
    unsigned keep;
    asm volatile("s_mov_b32 %0, m0\n\ts_mov_b32 m0, %3\n\ts_nop 0\n\t"
                 "global_load_lds_dwordx4 %1, %2 " GLDS_POL "\n\t"
                 "global_load_lds_dwordx4 %1, %2 offset:1024 " GLDS_POL "\n\t"
                 "global_load_lds_dwordx4 %1, %2 offset:2048 " GLDS_POL "\n\t"
                 "global_load_lds_dwordx4 %1, %2 offset:3072 " GLDS_POL "\n\t"
                 "s_mov_b32 m0, %0"
                 : "=&s"(keep) : "v"(voff), "s"(gbase), "s"(lds) : "memory");
}
__device__ __forceinline__ void glds1(const char* gbase, unsigned voff, unsigned lds) {
    unsigned keep;
    asm volatile("s_mov_b32 %0, m0\n\ts_mov_b32 m0, %3\n\ts_nop 0\n\t"
                 "global_load_lds_dwordx4 %1, %2\n\t"
                 "s_mov_b32 m0, %0"
                 : "=&s"(keep) : "v"(voff), "s"(gbase), "s"(lds) : "memory");
}
__device__ __forceinline__ void glds_flags(const unsigned* addr, unsigned lds) {
    unsigned keep;
    asm volatile("s_mov_b32 %0, m0\n\ts_mov_b32 m0, %2\n\ts_nop 0\n\t"
                 "global_load_lds_dword %1, off " POLL_POL "\n\t"
                 "s_mov_b32 m0, %0"
                 : "=&s"(keep) : "v"(addr), "s"(lds) : "memory");
}
__device__ __forceinline__ bool expired(u64 t0) { return __builtin_amdgcn_s_memrealtime() - t0 > DEADLINE; }
__device__ __forceinline__ unsigned lds_rd(volatile LAS unsigned* a) { return __builtin_amdgcn_readfirstlane(*a); }
__device__ __forceinline__ void lds_wait_ge(volatile LAS unsigned* a, unsigned v) {
    while ((int)(lds_rd(a) - v) < 0) __builtin_amdgcn_s_sleep(1);
    CFENCE;
}

__global__ void __launch_bounds__(NWAVE * 64, 4) fused_gemm_ar_q8(Params p) {
    extern __shared__ __attribute__((aligned(16))) char smem[];
    const int tid = threadIdx.x;
    const int lane = tid & 63;
    const int wave = __builtin_amdgcn_readfirstlane(tid >> 6);
    const int b = blockIdx.x;
    const int rank = p.rank;
    char* const myws = p.ws[rank];
    unsigned* const tmo = (unsigned*)(myws + OFF_TMO);
    volatile LAS unsigned* aux = (volatile LAS unsigned*)(smem + LDS_AUX);
    unsigned* const ctr = (unsigned*)(myws + OFF_CTR) + (b * 16 + wave);
    const int xcd = b & 7, idx = b >> 3;

    const int dd = COPY_MAP ? b >> 5 : b & 7, q = COPY_MAP ? b & 31 : b >> 3;
    const int dest = (rank + dd) & 7;
    if (wave == W_FLAG) {
        const unsigned seq = __builtin_amdgcn_readfirstlane(__hip_atomic_load(ctr, RLX, AGT) + 1u);
        unsigned* const prog = (unsigned*)(p.ws[dest] + (lane == 0 ? OFF_PROGX + (unsigned)q * 128u + (unsigned)rank * 4u
                                                                    : OFF_PROGW + (unsigned)q * 32u + (unsigned)rank * 4u));
        __builtin_amdgcn_s_barrier();
        CFENCE;
        if (lane == 0) __hip_atomic_store(ctr, seq, RLX, AGT);
        unsigned done = 0;
        while (done < NSUB) {
            lds_wait_ge(aux + A_SENT, done + 1u);
            done = lds_rd(aux + A_SENT);
            VMCNT(0);
            if (lane < 2) __hip_atomic_store(prog, seq * 32u + done, RLX, SYS);
        }
        return;
    }
    if (wave == W_FETCH) {
        const int di = dest >> 2, dj = dest & 3;
        const unsigned xrow = (unsigned)(di * 2048 + q * 64 + (lane >> 2));
        const unsigned xkq = (unsigned)((lane & 3) ^ ((4 - (lane >> 4)) & 3));
        const unsigned xo = xrow * (KC * 2) + xkq * 16u;
        const unsigned kk = (unsigned)(lane >> 4), k3 = (unsigned)((lane >> 2) & 1), k2 = (unsigned)(((lane >> 3) & 1) ^ (q & 1));
        const unsigned c4 = (unsigned)(lane & 3) ^ kk;
        const unsigned wk = kk | (k2 << 2) | (k3 << 3);
        const unsigned wo = wk * (NCOL * 2) + (unsigned)(dj * 1024 + q * 32) * 2u + c4 * 16u;
        const unsigned lb = (unsigned)(size_t)(LAS char*)smem + LDS_CPY;
        auto fetch = [&](int s) {
            const unsigned l0 = lb + (unsigned)(s % CG) * 6144u;
#pragma unroll
            for (int j = 0; j < 4; ++j) glds1(p.x, xo + (unsigned)(s * 64 + j * 16 * (KC * 2)), l0 + j * 1024u);
#pragma unroll
            for (int j = 0; j < 2; ++j) glds1(p.w, wo + (unsigned)((s * 32 + j * 16) * (NCOL * 2)), l0 + 4096u + j * 1024u);
        };
#pragma unroll
        for (int s = 0; s < CG; ++s) fetch(s);
        __builtin_amdgcn_s_barrier();
        CFENCE;
#pragma unroll
        for (int s = 0; s < NSUB; ++s) {
            constexpr int dummy = 0; (void)dummy;
            const int behind = (CG + s < NSUB ? CG + s : NSUB) - s - 1;
            if (behind == 4) VMCNT(24); else if (behind == 3) VMCNT(18); else if (behind == 2) VMCNT(12); else if (behind == 1) VMCNT(6); else VMCNT(0);
            if (lane == 0) aux[A_FETCHED] = (unsigned)(s + 1);
            if (s + CG < NSUB) { lds_wait_ge(aux + A_TAKEN, (unsigned)(s + 1)); fetch(s + CG); }
        }
        return;
    }
    if (wave == W_COPY) {
        __builtin_amdgcn_s_setprio(CPRIO);
        const unsigned seqv = __hip_atomic_load(ctr, RLX, AGT) + 1u;
        STAMP(8);
        char* const dws = p.ws[dest];
        auto rs = mk_rsrc(dws + OFF_STAGE + (size_t)rank * NSUB * UNIT, NSUB * UNIT);
        const unsigned ox = (unsigned)q * 4096u + (unsigned)lane * 16u;
        const unsigned ow = UNIT_X + (unsigned)q * 2048u + (unsigned)lane * 16u;
        const LAS char* const src = (const LAS char*)smem + LDS_CPY + (unsigned)lane * 16u;
        __builtin_amdgcn_s_barrier();
        CFENCE;
        const unsigned seq = __builtin_amdgcn_readfirstlane(seqv);
#pragma unroll
        for (int s = 0; s < NSUB; ++s) {
            lds_wait_ge(aux + A_FETCHED, (unsigned)(s + 1));
            v4u d[6];
#pragma unroll
            for (int j = 0; j < 6; ++j) d[j] = *(const LAS v4u*)(src + (s % CG) * 6144 + j * 1024);
            LGKM0;
            if (lane == 0) aux[A_TAKEN] = (unsigned)(s + 1);
#pragma unroll
            for (int j = 0; j < 6; ++j) {
                if (s > 0 || j >= COPY_D) asm volatile("s_waitcnt vmcnt(%0)" :: "n"(COPY_D - 1) : "memory");
                if (s > 0 && j == COPY_D - 1 && lane == 0) aux[A_SENT] = (unsigned)s;
                if (j < 4) __builtin_amdgcn_raw_buffer_store_b128(d[j], rs, ox + j * 1024 + s * UNIT, 0, 17);
                else __builtin_amdgcn_raw_buffer_store_b128(d[j], rs, ow + (j - 4) * 1024 + s * UNIT, 0, 17);
                if (s == 0 && j == 0) STAMP(9);
            }
        }
        VMCNT(0);
        if (lane == 0) {
            aux[A_SENT] = (unsigned)NSUB;
            __hip_atomic_store(ctr, seq, RLX, AGT);
        }
        STAMP(10);
        return;
    }

    const int tm = (xcd >> 1) * 8 + (idx & 7);
    const int tn = (xcd & 1) * 4 + (idx >> 3);
    const unsigned ldsbase = (unsigned)(size_t)(LAS char*)smem;
    if (tid < 64) aux[tid] = 0u;

    if (wave < 4) {
        const int wm = wave >> 1, wn = wave & 1;
        const int fm = lane & 15, fq = lane >> 4;
        v4f acc[2][4];
#pragma unroll
        for (int i = 0; i < 2; ++i)
#pragma unroll
            for (int j = 0; j < 4; ++j) acc[i][j] = v4f{0.f, 0.f, 0.f, 0.f};
        const unsigned xaddr = (unsigned)wm * 2048u + 16u * (unsigned)(4 * fm + (fq ^ ((4 - (fm >> 2)) & 3)));
        unsigned waddr;
        {
            const unsigned gg = (unsigned)fq, aa = (unsigned)((lane >> 2) & 3), bb = (unsigned)(lane & 3);
            const unsigned pp = bb >> 1, beta = bb & 1;
            const unsigned pos = ((2u * beta) ^ aa) | ((gg & 1u) << 2) | (pp << 3) | (aa << 4) | ((gg >> 1) << 6);
            waddr = 4096u + ((unsigned)wn * 2u + pp) * 2048u + 16u * pos + 8u * beta;
        }
        LGKM0;
        __builtin_amdgcn_s_barrier();
        CFENCE;
        if (wave == 0) STAMP(0);
        v8s xf[2][2], wf[2][4];
        auto rd = [&](int set, int slot) {
            const LAS char* sl = (const LAS char*)smem + (unsigned)slot * SLOT;
#pragma unroll
            for (int mt = 0; mt < 2; ++mt) xf[set][mt] = *(const LAS v8s*)(sl + xaddr + mt * 1024);
#pragma unroll
            for (int t = 0; t < 4; ++t) {
                const unsigned a0 = waddr ^ (unsigned)((t >> 1) << 4) ^ (unsigned)((t & 1) << 3);
                v4s lo = __builtin_amdgcn_ds_read_tr16_b64_v4i16((LAS v4s*)(sl + a0));
                v4s hi = __builtin_amdgcn_ds_read_tr16_b64_v4i16((LAS v4s*)(sl + (a0 ^ 128u)));
                wf[set][t] = __builtin_shufflevector(lo, hi, 0, 1, 2, 3, 4, 5, 6, 7);
            }
        };
        auto mm = [&](int set) {
#pragma unroll
            for (int t = 0; t < 4; ++t)
#pragma unroll
                for (int mt = 0; mt < 2; ++mt)
                    acc[mt][t] = __builtin_amdgcn_mfma_f32_16x16x32_bf16((v8bf)wf[set][t], (v8bf)xf[set][mt], acc[mt][t], 0, 0, 0);
        };
        auto step = [&](int set, int n, int nslot) {
            const bool more = n + 1 < NUNIT;
            const bool early = more && (int)(lds_rd(aux + A_FULL + nslot) - (unsigned)(n + 2)) >= 0;
            LGKM0;
            if (lane == 0) aux[A_CONS + wave] = (unsigned)(n + 1);
            CFENCE;
            if (early) {
                rd(set ^ 1, nslot);
                mm(set);
            } else {
                mm(set);
                if (more) { lds_wait_ge(aux + A_FULL + nslot, (unsigned)(n + 2)); rd(set ^ 1, nslot); }
            }
        };
        lds_wait_ge(aux + A_FULL, 1u);
        rd(0, 0);
        int slot = 0;
        for (int n = 0; n < NUNIT; n += 2) {
            step(0, n, slot + 1);
            slot = slot + 2 == NSLOT ? 0 : slot + 2;
            step(1, n + 1, slot);
        }
        if (wave == 0) STAMP(3);
        float mx = 0.f;
#pragma unroll
        for (int i = 0; i < 2; ++i)
#pragma unroll
            for (int j = 0; j < 4; ++j)
#pragma unroll
                for (int e = 0; e < 4; ++e) { acc[i][j][e] = fmaxf(acc[i][j][e], 0.f); mx = fmaxf(mx, acc[i][j][e]); }
#pragma unroll
        for (int o = 32; o > 0; o >>= 1) mx = fmaxf(mx, __shfl_xor(mx, o));
        if (lane == 0) {
            __hip_atomic_fetch_max((LAS unsigned*)(smem + LDS_AUX) + A_MAX, __float_as_uint(mx), RLX, WGS);
            CFENCE;
            __hip_atomic_fetch_add((LAS unsigned*)(smem + LDS_AUX) + A_CNT, 1u, RLX, WGS);
        }
        lds_wait_ge(aux + A_GO, 1u);
        if (wave == 0) STAMP(4);
        const float amax = __uint_as_float(aux[A_GM]);
        const bool dead = aux[A_DEAD] != 0u;
        const float inv = amax > 0.f ? 127.0f / amax : 0.f;
#pragma unroll
        for (int mt = 0; mt < 2; ++mt) {
            unsigned dw[4];
#pragma unroll
            for (int t = 0; t < 4; ++t) {
                unsigned r[4];
#pragma unroll
                for (int e = 0; e < 4; ++e) r[e] = __float_as_uint(fminf(__builtin_fmaf(acc[mt][t][e], inv, 12582912.0f), 12582912.0f + 127.0f));
                const unsigned lo = __builtin_amdgcn_perm(r[1], r[0], 0x0c0c0400u);
                const unsigned hi = __builtin_amdgcn_perm(r[3], r[2], 0x04000c0cu);
                dw[t] = lo | hi;
            }
            const bool odd = fq & 1;
            v4u qv;
            qv.x = odd ? dw[1] : dw[0];
            qv.y = odd ? dw[0] : dw[1];
            qv.z = odd ? dw[3] : dw[2];
            qv.w = odd ? dw[2] : dw[3];
            if (dead) qv = v4u{0x80808080u, 0x80808080u, 0x80808080u, 0x80808080u};
            *(LAS v4u*)(smem + (unsigned)(wm * 32 + mt * 16 + fm) * 144u + (unsigned)(wn * 64 + fq * 16)) = qv;
        }
        CFENCE;
        if (lane == 0) __hip_atomic_fetch_add((LAS unsigned*)(smem + LDS_AUX) + A_CNT, 1u, RLX, WGS);
        return;
    }

    const unsigned seqv = __hip_atomic_load(ctr, RLX, AGT) + 1u;
    const unsigned seq = __builtin_amdgcn_readfirstlane(seqv);
    if (lane == 0) __hip_atomic_store(ctr, seq, RLX, AGT);
    const u64 t0 = __builtin_amdgcn_s_memrealtime();

    if (wave == W_POLL) {
        const char* const px = myws + OFF_PROGX + (unsigned)tm * 128u;
        const char* const pw = myws + OFF_PROGW + (unsigned)tn * 128u;
        const unsigned pbase = seq * 32u;
        __builtin_amdgcn_s_waitcnt(0x0F70);
        __builtin_amdgcn_s_barrier();
        CFENCE;
        for (;;) {
            const u64 now = __builtin_amdgcn_s_memrealtime();
            v8u fx; v16u f0, f1;
            asm volatile("s_load_dwordx8 %0, %3, 0x0 glc\n\ts_load_dwordx16 %1, %4, 0x0 glc\n\ts_load_dwordx16 %2, %4, 0x40 glc\n\ts_waitcnt lgkmcnt(0)"
                         : "=&s"(fx), "=&s"(f0), "=&s"(f1) : "s"(px), "s"(pw) : "memory");
            int pv = 0, lo = NSUB;
#pragma unroll
            for (int c = 0; c < 8; ++c) {
                const int m = min(min(min((int)(f0[c] - pbase), (int)(f0[8 + c] - pbase)), min((int)(f1[c] - pbase), (int)(f1[8 + c] - pbase))), (int)(fx[c] - pbase));
                lo = min(lo, m);
                pv = lane == c ? m : pv;
            }
            bool stop = lo >= NSUB;
            if (!stop && now - t0 > 50000ull) {
                if (now - t0 > DEADLINE || __hip_atomic_load(tmo, RLX, AGT) == seq) {
                    __hip_atomic_store(tmo, seq, RLX, AGT);
                    pv = NSUB;
                    stop = true;
                }
            }
            if (lane < 8) aux[A_PRG + lane] = (unsigned)pv;
            if (stop) return;
            const u64 gap = lo < NSUB - FAST_LEFT ? GAP_SLOW : GAP_FAST;
            while (__builtin_amdgcn_s_memrealtime() - now < gap) __builtin_amdgcn_s_sleep(1);
        }
    }
    if (wave == W_SCHED) {
        const int csrc = (rank + lane) & 7;
        int nxt = lane < 8 ? 0 : NSUB;
        int asg = 0;
        __builtin_amdgcn_s_waitcnt(0x0F70);
        __builtin_amdgcn_s_barrier();
        CFENCE;
        while (asg < NUNIT) {
            const int prg = (int)aux[A_PRG + csrc];
            const v4u cs = *(const volatile LAS v4u*)(aux + A_CONS);
            const int lim = __builtin_amdgcn_readfirstlane(min(min(cs.x, cs.y), min(cs.z, cs.w))) + NSLOT;
            const int asg0 = asg;
            while (asg < lim) {
                const unsigned av = (unsigned)__ballot(prg > nxt);
                if (av == 0) break;
                const int c = __builtin_ctz(av);
                const int s = __builtin_amdgcn_readlane(nxt, c);
                if (lane == 0) aux[A_DESC + (asg & 15)] = (unsigned)(((rank + c) & 7) * NSUB + s) * UNIT;
                nxt = lane == c ? nxt + 1 : nxt;
                ++asg;
            }
            if (asg != asg0) { CFENCE; if (lane == 0) aux[A_ASG] = (unsigned)asg; }
            else __builtin_amdgcn_s_sleep(1);
        }
        lds_wait_ge(aux + A_CNT, 4u);
        if (lane == 0) __hip_atomic_store((u64*)(myws + OFF_WGMAX + (unsigned)b * 8u), ((u64)seq << 32) | aux[A_MAX], RLX, AGT);
        if (b == 0) {
            auto rg = mk_rsrc(myws + OFF_WGMAX, NWG * 8);
            unsigned mx;
            for (;;) {
                v4u g0 = __builtin_amdgcn_raw_buffer_load_b128(rg, (unsigned)lane * 32u, 0, 17);
                v4u g1 = __builtin_amdgcn_raw_buffer_load_b128(rg, (unsigned)lane * 32u + 16u, 0, 17);
                unsigned tm_ = __hip_atomic_load(tmo, RLX, AGT);
                bool ok = g0.y == seq && g0.w == seq && g1.y == seq && g1.w == seq;
                mx = max(max(g0.x, g0.z), max(g1.x, g1.z));
                if (__all(ok)) break;
                if (tm_ == seq || expired(t0)) { __hip_atomic_store(tmo, seq, RLX, AGT); break; }
            }
#pragma unroll
            for (int o = 32; o > 0; o >>= 1) mx = max(mx, (unsigned)__shfl_xor((int)mx, o));
            char* dst = p.ws[lane >> 3] + OFF_RMAX + (unsigned)(lane & 7) * 128u + (unsigned)rank * 8u;
            __hip_atomic_store((u64*)dst, ((u64)seq << 32) | mx, RLX, SYS);
            STAMP(7);
        }
        const u64* ra = lane < 8 ? (const u64*)(myws + OFF_RMAX + (unsigned)xcd * 128u + (unsigned)lane * 8u) : (const u64*)tmo;
        unsigned gm, xd;
        for (;;) {
            const u64 v = __hip_atomic_load(ra, RLX, SYS);
            xd = __builtin_amdgcn_readlane((unsigned)v, 8) == seq;
            gm = lane < 8 ? (unsigned)v : 0u;
            if (__all(lane >= 8 || (unsigned)(v >> 32) == seq)) break;
            if (xd || expired(t0)) { __hip_atomic_store(tmo, seq, RLX, AGT); xd = 1; break; }
            __builtin_amdgcn_s_sleep(1);
        }
        gm = max(gm, (unsigned)__shfl_xor((int)gm, 1));
        gm = max(gm, (unsigned)__shfl_xor((int)gm, 2));
        gm = max(gm, (unsigned)__shfl_xor((int)gm, 4));
        if (lane == 0) { aux[A_GM] = gm; aux[A_DEAD] = xd; CFENCE; aux[A_GO] = 1u; }
        return;
    }

    {
        const int l = wave - 4;
        const char* const stage = myws + OFF_STAGE;
        const unsigned vx = (unsigned)tm * 4096u + (unsigned)lane * 16u;
        const unsigned vw = UNIT_X + (unsigned)tn * 8192u + (unsigned)lane * 16u;
        __builtin_amdgcn_s_waitcnt(0x0F70);
        __builtin_amdgcn_s_barrier();
        CFENCE;
        int slot = l;
        for (int n = l; n < NUNIT; n += NL) {
            lds_wait_ge(aux + A_ASG, (unsigned)(n + 1));
            const char* gb = stage + lds_rd(aux + A_DESC + (n & 15));
            const unsigned ld = ldsbase + (unsigned)slot * SLOT;
            glds4(gb, vx, ld);
            glds4(gb, vw, ld + 4096u);
            glds4(gb + 4096, vw, ld + 8192u);
            VMCNT(0);
            if (lane == 0) aux[A_FULL + slot] = (unsigned)(n + 1);
            slot += NL;
            if (slot >= NSLOT) slot -= NSLOT;
        }
        if (l < 2) {
            lds_wait_ge(aux + A_CNT, 8u);
            const unsigned ooff = (unsigned)((rank >> 2) * 2048 + tm * 64 + (lane >> 3)) * NCOL + (unsigned)((rank & 3) * 1024 + tn * 128 + (lane & 7) * 16);
            const LAS char* src = (const LAS char*)smem + (unsigned)(lane >> 3) * 144u + (unsigned)(lane & 7) * 16u;
            v4u qv[8];
#pragma unroll
            for (int i = 0; i < 8; ++i) qv[i] = *(const LAS v4u*)(src + i * 8 * 144);
            int prev = rank;
#pragma unroll
            for (int k = 0; k < 4; ++k) {
                const int dst = (rank + ((l + 2 * k + idx) & 7)) & 7;
                auto ro = mk_rsrc(p.out[dst], (unsigned)NCOL * 4096u);
#pragma unroll
                for (int i = 0; i < 8; ++i) {
                    __builtin_amdgcn_raw_buffer_store_b128(qv[i], ro, ooff + i * 8 * NCOL, 0, 17);
                    asm volatile("s_waitcnt vmcnt(%0)" :: "n"(P2_D - 1) : "memory");
                    if (i == P2_D - 2 + (P2_D == 1) && k > 0 && prev != rank && lane == 0)
                        __hip_atomic_store((unsigned*)(p.ws[prev] + OFF_DONE + (unsigned)rank * 1024u + (unsigned)b * 4u), seq, RLX, SYS);
                }
                prev = dst;
            }
            VMCNT(0);
            if (prev != rank && lane == 0)
                __hip_atomic_store((unsigned*)(p.ws[prev] + OFF_DONE + (unsigned)rank * 1024u + (unsigned)b * 4u), seq, RLX, SYS);
            if (l == 0) STAMP(5);
        } else if (b == 0 && l < 6) {
            auto rd = mk_rsrc(myws + OFF_DONE, NR * NWG * 4);
            const int s0 = (l - 2) * 2, s1 = s0 + 1;
            for (;;) {
                v4u g0 = __builtin_amdgcn_raw_buffer_load_b128(rd, (unsigned)s0 * 1024u + (unsigned)lane * 16u, 0, 17);
                v4u g1 = __builtin_amdgcn_raw_buffer_load_b128(rd, (unsigned)s1 * 1024u + (unsigned)lane * 16u, 0, 17);
                unsigned tm_ = __hip_atomic_load(tmo, RLX, AGT);
                bool ok0 = s0 == rank || ((int)(g0.x - seq) >= 0 && (int)(g0.y - seq) >= 0 && (int)(g0.z - seq) >= 0 && (int)(g0.w - seq) >= 0);
                bool ok1 = s1 == rank || ((int)(g1.x - seq) >= 0 && (int)(g1.y - seq) >= 0 && (int)(g1.z - seq) >= 0 && (int)(g1.w - seq) >= 0);
                if (__all(ok0 && ok1)) break;
                if (tm_ == seq || expired(t0)) {
                    __hip_atomic_store(tmo, seq, RLX, AGT);
                    if (lane == 0) p.out[rank][(size_t)(rank >> 2) * 2048 * NCOL + (rank & 3) * 1024] = (char)-128;
                    break;
                }
                __builtin_amdgcn_s_sleep(2);
            }
            STAMP(9 + l);
        }
    }
}

}

#define HIP_CHECK(expr) do { hipError_t _e = (expr); if (_e != hipSuccess) { \
    fprintf(stderr, "HIP error %s at %s:%d\n", hipGetErrorString(_e), __FILE__, __LINE__); \
    return; } } while (0)

extern "C" void kernel_launch(int rank, int nranks, ncclComm_t comm, hipStream_t stream,
                              void* const* d_in, const int* in_sizes, int n_in,
                              void* d_out, int out_size, const int* in_dtypes, int out_dtype,
                              void* const* d_out_all, void* const* d_ws_all, size_t ws_bytes) {
    (void)comm; (void)in_sizes; (void)n_in; (void)d_out; (void)in_dtypes; (void)out_dtype;
    if (nranks != NR || (unsigned)rank >= (unsigned)NR || ws_bytes < WS_NEED || out_size != NCOL * 4096) {
        fprintf(stderr, "unexpected problem geometry\n");
        return;
    }
    static bool attr_done[NR] = {false};
    if (!attr_done[rank]) {
        HIP_CHECK(hipFuncSetAttribute((const void*)fused_gemm_ar_q8, hipFuncAttributeMaxDynamicSharedMemorySize, LDS_BYTES + 256));
        attr_done[rank] = true;
    }
    Params p;
    for (int r = 0; r < NR; ++r) { p.ws[r] = (char*)d_ws_all[r]; p.out[r] = (char*)d_out_all[r]; }
    p.x = (const char*)d_in[0];
    p.w = (const char*)d_in[1];
    p.rank = rank;
    hipLaunchKernelGGL(fused_gemm_ar_q8, dim3(NWG), dim3(NWAVE * 64), LDS_BYTES + 256, stream, p);
    HIP_CHECK(hipGetLastError());
}
```

```cpp
#include <hip/hip_runtime.h>
#include <rccl/rccl.h>
#include <cstdio>
#include <cstdint>

#define COPY_D 3
#define PACE 342
#define GAP_SLOW 100
#define GAP_FAST 30
#define FAST_LEFT 3
#define OUT_AUX 16
#define PREP 2
#define VPOLL 0
#define GAP_VEC 15
#define VPACE 65
#define CMP_PRIO 2
#define MPACE 65
#define FSTEP 2u
#define FINE_FROM 12u
#define P2_D 3

namespace {

typedef short v4s __attribute__((ext_vector_type(4)));
typedef short v8s __attribute__((ext_vector_type(8)));
typedef __bf16 v8bf __attribute__((ext_vector_type(8)));
typedef float v4f __attribute__((ext_vector_type(4)));
typedef unsigned v4u __attribute__((ext_vector_type(4)));
typedef unsigned v8u __attribute__((ext_vector_type(8)));
typedef unsigned v16u __attribute__((ext_vector_type(16)));
typedef unsigned long long u64;
#define LAS __attribute__((address_space(3)))

constexpr int NR = 8;
constexpr int NCOL = 4096;
constexpr int KC = 512;
constexpr int NSUB = 16;
constexpr int NUNIT = NR * NSUB;
constexpr int NWG = 256;
constexpr int NL = 7;
constexpr int W_SCHED = 4 + NL, W_COPY = 5 + NL  , W_FETCH = 7 + NL, W_POLL = 8 + NL, NWAVE = 9 + NL;
constexpr unsigned UNIT_X = 2048u * 32u * 2u;
constexpr unsigned UNIT_W = 32u * 1024u * 2u;
constexpr unsigned UNIT = UNIT_X + UNIT_W;
constexpr unsigned OFF_CTR = 0;
constexpr unsigned OFF_TMO = 32768;
constexpr unsigned OFF_WGMAX = 40960;
constexpr unsigned OFF_RMAX = 45056;
constexpr unsigned OFF_GMAX = 49152;
constexpr unsigned OFF_DIAG = 65536;
constexpr unsigned OFF_PROG = 262144;
constexpr unsigned OFF_P2F = 294912;
constexpr unsigned OFF_STAGE = 1u << 20;
constexpr unsigned P2_PITCH = 10240;
constexpr unsigned OFF_P2 = OFF_STAGE + NR * NSUB * UNIT;
constexpr unsigned WS_NEED = OFF_P2 + NR * NWG * P2_PITCH;
constexpr unsigned SLOT = 12288;
constexpr int NSLOT = 10;
constexpr unsigned LDS_AUX = NSLOT * SLOT;
constexpr unsigned LDS_CPY = LDS_AUX + 256;
constexpr int CG = 5;
constexpr unsigned L_MASK = LDS_CPY + CG * 6144;
constexpr unsigned LDS_BYTES = L_MASK + 1024;
constexpr unsigned L_STREAM = 0, L_PACK = 36864, L_DUMMY = 49152;
constexpr int A_ASG = 0, A_CDONE = 1, A_SENT = 2, A_CONS = 4, A_MAX = 8, A_CNT = 9, A_CNT2 = 10, A_CNT3 = 11, A_GM = 12, A_DEAD = 13,
              A_GO = 14, A_CNT1 = 15, A_DESC = 16, A_FULL = 32, A_TOT = 44, A_FETCHED = 48, A_MS = 49, A_TAKEN = 50, A_P2 = 52, A_TGO = 54,
              A_LEAD = 55, A_PRG = 56;
constexpr u64 DEADLINE = 100000000ull;

struct Params {
    char* ws[NR];
    char* out[NR];
    const char* x;
    const char* w;
    int rank;
};

#define SYS __HIP_MEMORY_SCOPE_SYSTEM
#define AGT __HIP_MEMORY_SCOPE_AGENT
#define WGS __HIP_MEMORY_SCOPE_WORKGROUP
#define RLX __ATOMIC_RELAXED
#define VMCNT(n) asm volatile("s_waitcnt vmcnt(" #n ")" ::: "memory")
#define VMCNTN(n) asm volatile("s_waitcnt vmcnt(%0)" :: "n"(n) : "memory")
#define VMCNT0_SEEN __builtin_amdgcn_s_waitcnt(0x0F70)
#define LGKM0 asm volatile("s_waitcnt lgkmcnt(0)" ::: "memory")
#define CFENCE asm volatile("" ::: "memory")
#define START_BARRIER do { __builtin_amdgcn_s_barrier(); CFENCE; } while (0)

#define STAMP(slot) do { } while (0)

__device__ __forceinline__ auto mk_rsrc(const void* base, unsigned bytes) {
    return __builtin_amdgcn_make_buffer_rsrc(const_cast<void*>(base), 0, bytes, 0x00020000);
}
__device__ __forceinline__ void glds4(const char* gbase, unsigned voff, unsigned lds) {
    unsigned keep;
    asm volatile("s_mov_b32 %0, m0\n\ts_mov_b32 m0, %3\n\ts_nop 0\n\t"
                 "global_load_lds_dwordx4 %1, %2 sc1\n\t"
                 "global_load_lds_dwordx4 %1, %2 offset:1024 sc1\n\t"
                 "global_load_lds_dwordx4 %1, %2 offset:2048 sc1\n\t"
                 "global_load_lds_dwordx4 %1, %2 offset:3072 sc1\n\t"
                 "s_mov_b32 m0, %0"
                 : "=&s"(keep) : "v"(voff), "s"(gbase), "s"(lds) : "memory");
}
__device__ __forceinline__ void glds1(const char* gbase, unsigned voff, unsigned lds) {
    unsigned keep;
    asm volatile("s_mov_b32 %0, m0\n\ts_mov_b32 m0, %3\n\ts_nop 0\n\t"
                 "global_load_lds_dwordx4 %1, %2\n\t"
                 "s_mov_b32 m0, %0"
                 : "=&s"(keep) : "v"(voff), "s"(gbase), "s"(lds) : "memory");
}
__device__ __forceinline__ bool expired(u64 t0) { return __builtin_amdgcn_s_memrealtime() - t0 > DEADLINE; }
__device__ __forceinline__ unsigned lds_rd(volatile LAS unsigned* a) { return __builtin_amdgcn_readfirstlane(*a); }
__device__ __forceinline__ void lds_wait_ge(volatile LAS unsigned* a, unsigned v) {
    while ((int)(lds_rd(a) - v) < 0) __builtin_amdgcn_s_sleep(1);
    CFENCE;
}
__device__ __forceinline__ void lds_add(volatile LAS unsigned* a) {
    __hip_atomic_fetch_add((LAS unsigned*)a, 1u, RLX, WGS);
}
__device__ __forceinline__ unsigned wave_max(unsigned v) {
    v = max(v, (unsigned)__builtin_amdgcn_update_dpp(0, (int)v, 0x111, 0xf, 0xf, false));
    v = max(v, (unsigned)__builtin_amdgcn_update_dpp(0, (int)v, 0x112, 0xf, 0xf, false));
    v = max(v, (unsigned)__builtin_amdgcn_update_dpp(0, (int)v, 0x114, 0xf, 0xf, false));
    v = max(v, (unsigned)__builtin_amdgcn_update_dpp(0, (int)v, 0x118, 0xf, 0xf, false));
    return max(max((unsigned)__builtin_amdgcn_readlane((int)v, 15), (unsigned)__builtin_amdgcn_readlane((int)v, 31)),
               max((unsigned)__builtin_amdgcn_readlane((int)v, 47), (unsigned)__builtin_amdgcn_readlane((int)v, 63)));
}
__device__ __forceinline__ unsigned lane_prefix(unsigned c, unsigned* total) {
    unsigned pre = 0, tot = 0;
#pragma unroll
    for (int k = 0; k < 6; ++k) {
        const u64 m = __ballot((c >> k) & 1u);
        pre += __builtin_amdgcn_mbcnt_hi((unsigned)(m >> 32), __builtin_amdgcn_mbcnt_lo((unsigned)m, 0u)) << k;
        tot += (unsigned)__builtin_popcountll(m) << k;
    }
    *total = tot;
    return pre;
}
__device__ __forceinline__ unsigned quant4(v4f y, float inv) {
    unsigned r[4];
#pragma unroll
    for (int e = 0; e < 4; ++e) r[e] = __float_as_uint(fminf(__builtin_fmaf(y[e], inv, 12582912.0f), 12582912.0f + 127.0f));
    return __builtin_amdgcn_perm(r[1], r[0], 0x0c0c0400u) | __builtin_amdgcn_perm(r[3], r[2], 0x04000c0cu);
}
__device__ __forceinline__ bool wait_word(const char* addr, unsigned v, unsigned* tmo, unsigned seq, u64 t0) {
    for (;;) {
        unsigned f;
        asm volatile("s_load_dword %0, %1, 0x0 glc\n\ts_waitcnt lgkmcnt(0)" : "=s"(f) : "s"(addr) : "memory");
        if ((int)(f - v) >= 0) return true;
        const u64 now = __builtin_amdgcn_s_memrealtime();
        if (now - t0 > 50000ull) {
            if (now - t0 > DEADLINE || __hip_atomic_load(tmo, RLX, AGT) == seq) {
                __hip_atomic_store(tmo, seq, RLX, AGT);
                return false;
            }
        }
        __builtin_amdgcn_s_sleep(2);
    }
}

__global__ void __launch_bounds__(NWAVE * 64, 4) fused_gemm_ar_q8(Params p) {
    extern __shared__ __attribute__((aligned(16))) char smem[];
    const int tid = threadIdx.x;
    const int lane = tid & 63;
    const int wave = __builtin_amdgcn_readfirstlane(tid >> 6);
    const int b = blockIdx.x;
    const int rank = p.rank;
    char* const myws = p.ws[rank];
    unsigned* const tmo = (unsigned*)(myws + OFF_TMO);
    volatile LAS unsigned* aux = (volatile LAS unsigned*)(smem + LDS_AUX);
    const unsigned ldsbase = (unsigned)(size_t)(LAS char*)smem;
    const int xcd = b & 7, idx = b >> 3;
    const int tm = (xcd >> 1) * 8 + (idx & 7);
    const int tn = (xcd & 1) * 4 + (idx >> 3);
    const int fm = lane & 15, fq = lane >> 4;

    if (wave < 4) {
        if (tid < 64) aux[tid] = 0u;
        const int wm = wave >> 1, wn = wave & 1;
        v4f acc[2][4];
#pragma unroll
        for (int i = 0; i < 2; ++i)
#pragma unroll
            for (int j = 0; j < 4; ++j) acc[i][j] = v4f{0.f, 0.f, 0.f, 0.f};
        const unsigned xaddr = (unsigned)wm * 2048u + 16u * (unsigned)(4 * fm + (fq ^ ((4 - (fm >> 2)) & 3)));
        unsigned waddr;
        {
            const unsigned gg = (unsigned)fq, aa = (unsigned)((lane >> 2) & 3), bb = (unsigned)(lane & 3);
            const unsigned pp = bb >> 1, beta = bb & 1;
            const unsigned pos = ((2u * beta) ^ aa) | ((gg & 1u) << 2) | (pp << 3) | (aa << 4) | ((gg >> 1) << 6);
            waddr = 4096u + ((unsigned)wn * 2u + pp) * 2048u + 16u * pos + 8u * beta;
        }
        LGKM0;
        START_BARRIER;
        if (wave == 0) STAMP(0);
        {
            v8s xf[2][2], wf[2][4];
            auto rd = [&](int set, int slot) {
                const LAS char* sl = (const LAS char*)smem + (unsigned)slot * SLOT;
#pragma unroll
                for (int mt = 0; mt < 2; ++mt) xf[set][mt] = *(const LAS v8s*)(sl + xaddr + mt * 1024);
#pragma unroll
                for (int t = 0; t < 4; ++t) {
                    const unsigned a0 = waddr ^ (unsigned)((t >> 1) << 4) ^ (unsigned)((t & 1) << 3);
                    v4s lo = __builtin_amdgcn_ds_read_tr16_b64_v4i16((LAS v4s*)(sl + a0));
                    v4s hi = __builtin_amdgcn_ds_read_tr16_b64_v4i16((LAS v4s*)(sl + (a0 ^ 128u)));
                    wf[set][t] = __builtin_shufflevector(lo, hi, 0, 1, 2, 3, 4, 5, 6, 7);
                }
            };
            auto mm = [&](int set) {
#pragma unroll
                for (int t = 0; t < 4; ++t)
#pragma unroll
                    for (int mt = 0; mt < 2; ++mt)
                        acc[mt][t] = __builtin_amdgcn_mfma_f32_16x16x32_bf16((v8bf)wf[set][t], (v8bf)xf[set][mt], acc[mt][t], 0, 0, 0);
            };
            auto step = [&](int set, int n, int nslot) {
                LGKM0;
                if (lane == 0) aux[A_CONS + wave] = (unsigned)(n + 1);
                const unsigned f = aux[A_FULL + nslot];
                CFENCE;
                rd(set ^ 1, nslot);
                __builtin_amdgcn_sched_barrier(0);
                mm(set);
                __builtin_amdgcn_sched_barrier(0);
                if (n + 1 < NUNIT && (int)(__builtin_amdgcn_readfirstlane(f) - (unsigned)(n + 2)) < 0) {
                    lds_wait_ge(aux + A_FULL + nslot, (unsigned)(n + 2));
                    rd(set ^ 1, nslot);
                }
            };
            __builtin_amdgcn_s_setprio(CMP_PRIO);
            lds_wait_ge(aux + A_FULL, 1u);
            rd(0, 0);
            int slot = 0;
            for (int n = 0; n < NUNIT; n += 2) {
                step(0, n, slot + 1);
                slot = slot + 2 == NSLOT ? 0 : slot + 2;
                step(1, n + 1, slot);
            }
        }
        if (wave == 0) STAMP(3);
        float mx = 0.f;
        unsigned m32 = 0;
#pragma unroll
        for (int i = 0; i < 2; ++i)
#pragma unroll
            for (int j = 0; j < 4; ++j)
#pragma unroll
                for (int e = 0; e < 4; ++e) {
                    acc[i][j][e] = fmaxf(acc[i][j][e], 0.f);
                    mx = fmaxf(mx, acc[i][j][e]);
                    m32 |= acc[i][j][e] > 0.f ? 1u << (i * 16 + j * 4 + e) : 0u;
                }
        *(LAS unsigned*)(smem + L_MASK + (unsigned)tid * 4u) = m32;
        const unsigned wmx = wave_max(__float_as_uint(mx));
        CFENCE;
        if (lane == 0) {
            __hip_atomic_fetch_max((LAS unsigned*)(aux + A_MAX), wmx, RLX, WGS);
            CFENCE;
            lds_add(aux + A_CNT);
        }
        unsigned tot;
        const unsigned pre = lane_prefix((unsigned)__builtin_popcount(m32), &tot);
        if (lane == 0) { aux[A_TOT + wave] = tot; CFENCE; lds_add(aux + A_CNT1); }
        lds_wait_ge(aux + A_CNT1, 4u);
        const v4u tt = *(const volatile LAS v4u*)(aux + A_TOT);
        const unsigned T = __builtin_amdgcn_readfirstlane(tt.x + tt.y + tt.z + tt.w);
        {
            unsigned a = L_STREAM + 4u * (pre + (wave > 0 ? tt.x : 0u) + (wave > 1 ? tt.y : 0u) + (wave > 2 ? tt.z : 0u));
            const unsigned dummy = L_DUMMY + (unsigned)tid * 4u;
#pragma unroll
            for (int i = 0; i < 2; ++i)
#pragma unroll
                for (int j = 0; j < 4; ++j)
#pragma unroll
                    for (int e = 0; e < 4; ++e) {
                        const bool on = acc[i][j][e] > 0.f;
                        *(LAS float*)(smem + (on ? a : dummy)) = acc[i][j][e];
                        a += on ? 4u : 0u;
                    }
        }
        CFENCE;
        if (lane == 0) lds_add(aux + A_CNT2);
        lds_wait_ge(aux + A_CNT2, 4u);
        v4f sv[2][4];
#pragma unroll
        for (int h = 0; h < 2; ++h)
#pragma unroll
            for (int j = 0; j < 4; ++j)
                sv[h][j] = *(const LAS v4f*)(smem + L_STREAM + (unsigned)((wave + 4 * h) * 4096 + lane * 64 + j * 16));
        lds_wait_ge(aux + A_GO, 1u);
        if (wave == 0) STAMP(4);
        const float amax = __uint_as_float(aux[A_GM]);
        const bool dead = aux[A_DEAD] != 0u;
        const float inv = amax > 0.f ? 127.0f / amax : 0.f;
        const v4u bad = {0x80808080u, 0x80808080u, 0x80808080u, 0x80808080u};
#pragma unroll
        for (int h = 0; h < 2; ++h) {
            if (h == 0 || T > 4096u) {
                v4u qv = {quant4(sv[h][0], inv), quant4(sv[h][1], inv), quant4(sv[h][2], inv), quant4(sv[h][3], inv)};
                if (dead) qv = bad;
                *(LAS v4u*)(smem + L_PACK + (unsigned)((wave + 4 * h) * 1024 + lane * 16)) = qv;
            }
        }
        CFENCE;
        if (lane == 0) lds_add(aux + A_CNT3);
        auto ro = mk_rsrc(p.out[rank], (unsigned)NCOL * 4096u);
        const unsigned ooff = (unsigned)((rank >> 2) * 2048 + tm * 64 + wm * 32 + fm) * NCOL + (unsigned)((rank & 3) * 1024 + tn * 128 + wn * 64 + fq * 16);
#pragma unroll
        for (int mt = 0; mt < 2; ++mt) {
            const unsigned d0 = quant4(acc[mt][0], inv), d1 = quant4(acc[mt][1], inv), d2 = quant4(acc[mt][2], inv), d3 = quant4(acc[mt][3], inv);
            const bool odd = fq & 1;
            v4u qv = {odd ? d1 : d0, odd ? d0 : d1, odd ? d3 : d2, odd ? d2 : d3};
            if (dead) qv = bad;
            __builtin_amdgcn_raw_buffer_store_b128(qv, ro, ooff + mt * 16 * NCOL, 0, OUT_AUX);
        }
        return;
    }

    unsigned* const ctr = (unsigned*)(myws + OFF_CTR) + (b * 16 + wave);
    const unsigned seqv = __hip_atomic_load(ctr, RLX, AGT) + 1u;
    const int q = b & 31;
    const int dest = (rank + (b >> 5)) & 7;

    unsigned* const prog = (unsigned*)(p.ws[dest] + OFF_PROG + (unsigned)((lane & (PREP - 1)) * 32 + q) * 128u + (unsigned)rank * 4u);

    if (wave == W_COPY || wave == W_COPY + 1) {
        const int st = wave - W_COPY;
        __builtin_amdgcn_s_setprio(3);
        if (st == 0) STAMP(8);
        {
            auto rs = mk_rsrc(p.ws[dest] + OFF_STAGE + (size_t)rank * NSUB * UNIT, NSUB * UNIT);
            const unsigned ox = (unsigned)q * 4096u + (unsigned)(st * 1024 + lane * 16);
            const unsigned ow = UNIT_X + (unsigned)q * 2048u + (unsigned)(st * 1024 + lane * 16);
            const LAS char* const src = (const LAS char*)smem + LDS_CPY + (unsigned)(st * 1024 + lane * 16);
            START_BARRIER;
            const unsigned tc = (unsigned)__builtin_amdgcn_s_memrealtime();
            const unsigned pace = dest != rank ? PACE : 0u;
            int grp = 0;
            for (int s = 0; s < NSUB; ++s) {
                lds_wait_ge(aux + A_FETCHED, (unsigned)(s + 1));
                v4u d[3];
#pragma unroll
                for (int j = 0; j < 3; ++j) d[j] = *(const LAS v4u*)(src + grp * 6144 + j * 2048);
                grp = grp == CG - 1 ? 0 : grp + 1;
                LGKM0;
                if (lane == 0) aux[A_TAKEN + st] = (unsigned)(s + 1);
#pragma unroll
                for (int j = 0; j < 3; ++j) {
                    VMCNT(1);
                    if (j == 1 && lane == 0) aux[A_SENT + st] = (unsigned)s;
                    const unsigned due = (unsigned)(s * 6 + j * 2 + st) * pace / 6u;
                    while ((unsigned)__builtin_amdgcn_s_memrealtime() - tc < due) __builtin_amdgcn_s_sleep(1);
                    __builtin_amdgcn_raw_buffer_store_b128(d[j], rs, (j < 2 ? ox + j * 2048 : ow) + (unsigned)s * UNIT, 0, 17);
                    if (s == 0 && j == 0 && st == 0) STAMP(9);
                }
            }
            VMCNT(0);
        }
        const unsigned seq = __builtin_amdgcn_readfirstlane(seqv);
        {
            unsigned old = 0;
            if (lane == 0) old = __hip_atomic_fetch_add((LAS unsigned*)(aux + A_CDONE), 1u, RLX, WGS);
            if (__builtin_amdgcn_readfirstlane(old) == 1u && lane < PREP) __hip_atomic_fetch_max(prog, seq * 32u + (unsigned)NSUB, RLX, SYS);
        }
        if (lane == 0) __hip_atomic_store(ctr, seq, RLX, AGT);
        STAMP(10);
        const unsigned poff = OFF_P2 + (unsigned)(rank * NWG + b) * P2_PITCH;
        const int rot = b % 7;
        lds_wait_ge(aux + A_CNT, 4u);
        {
            const v4u mk = *(const LAS v4u*)(smem + L_MASK + (unsigned)lane * 16u);
            if (b == 0) lds_wait_ge(aux + A_LEAD, 1u);
            const unsigned tm0 = (unsigned)__builtin_amdgcn_s_memrealtime();
            for (int k = st; k < 7; k += 2) {
                const int dst = (rank + 1 + (k + rot) % 7) & 7;
                VMCNT(1);
                if (b != 0) while ((unsigned)__builtin_amdgcn_s_memrealtime() - tm0 < (unsigned)k * MPACE) __builtin_amdgcn_s_sleep(1);
                __builtin_amdgcn_raw_buffer_store_b128(mk, mk_rsrc(p.ws[dst] + poff, 1024u), (unsigned)lane * 16u, 0, 17);
            }
        }
        if (st == 0) STAMP(11);
        int last = -1, pub = -1;
        if (lds_rd(aux + A_CNT3) < 4u) {
            VMCNT(0);
            if (lane == 0) lds_add(aux + A_MS);
            pub = 0;
            lds_wait_ge(aux + A_CNT3, 4u);
        }
        CFENCE;
        const v4u tt = *(const volatile LAS v4u*)(aux + A_TOT);
        const unsigned Tr = (__builtin_amdgcn_readfirstlane(tt.x + tt.y + tt.z + tt.w) + 63u) & ~63u;
        const int np = (int)((Tr + 1023u) >> 10);
        const unsigned ps = np ? (((Tr >> 6) + (unsigned)np - 1u) / (unsigned)np) << 6 : 0u;
        v4u pv[8];
#pragma unroll
        for (int j = 0; j < 8; ++j) pv[j] = *(const LAS v4u*)(smem + L_PACK + (unsigned)j * ps + (unsigned)lane * 16u);
        const unsigned tv = lds_rd(aux + A_TGO);
        for (int k = 0; k < 7; ++k) {
            char* const dp = p.ws[(rank + 1 + (k + rot) % 7) & 7] + poff + 1024u;
#pragma unroll
            for (int j = 0; j < 8; ++j) {
                if (j < np && ((k * np + j) & 1) == st) {
                    VMCNT(1);
                    if (last > pub) {
                        if (lane == 0) { if (pub < 0) lds_add(aux + A_MS); aux[A_P2 + st] = (unsigned)last; }
                        pub = last;
                    }
                    const unsigned due = (((unsigned)k * Tr + (unsigned)j * ps) * VPACE) >> 10;
                    while ((unsigned)__builtin_amdgcn_s_memrealtime() - tv < due) __builtin_amdgcn_s_sleep(1);
                    __builtin_amdgcn_raw_buffer_store_b128(pv[j], mk_rsrc(dp + (unsigned)j * ps, min(ps, Tr - (unsigned)j * ps)), (unsigned)lane * 16u, 0, 17);
                    last = k;
                }
            }
        }
        VMCNT(0);
        if (lane == 0) { if (pub < 0) lds_add(aux + A_MS); aux[A_P2 + st] = 7u; }
        STAMP(5);
        return;
    }

    if (wave == W_FETCH) {
        const int di = dest >> 2, dj = dest & 3;
        const unsigned xo = (unsigned)(di * 2048 + q * 64 + (lane >> 2)) * (KC * 2) + (unsigned)((lane & 3) ^ ((4 - (lane >> 4)) & 3)) * 16u;
        const unsigned kk = (unsigned)(lane >> 4), k3 = (unsigned)((lane >> 2) & 1), k2 = (unsigned)(((lane >> 3) & 1) ^ (q & 1));
        const unsigned wo = (kk | (k2 << 2) | (k3 << 3)) * (NCOL * 2) + (unsigned)(dj * 1024 + q * 32) * 2u + ((unsigned)(lane & 3) ^ kk) * 16u;
        auto fetch = [&](int s) {
            const unsigned l0 = ldsbase + LDS_CPY + (unsigned)(s % CG) * 6144u;
#pragma unroll
            for (int j = 0; j < 4; ++j) glds1(p.x, xo + (unsigned)(s * 64 + j * 16 * (KC * 2)), l0 + j * 1024u);
#pragma unroll
            for (int j = 0; j < 2; ++j) glds1(p.w, wo + (unsigned)((s * 32 + j * 16) * (NCOL * 2)), l0 + 4096u + j * 1024u);
        };
#pragma unroll
        for (int s = 0; s < CG; ++s) fetch(s);
        START_BARRIER;
#pragma unroll
        for (int s = 0; s < NSUB; ++s) {
            VMCNTN(6 * ((CG + s < NSUB ? CG + s : NSUB) - s - 1));
            if (lane == 0) aux[A_FETCHED] = (unsigned)(s + 1);
            if (s + CG < NSUB) { lds_wait_ge(aux + A_TAKEN, (unsigned)(s + 1)); lds_wait_ge(aux + A_TAKEN + 1, (unsigned)(s + 1)); fetch(s + CG); }
        }
        VMCNT0_SEEN;
        const unsigned seq = __builtin_amdgcn_readfirstlane(seqv);
        if (lane == 0) __hip_atomic_store(ctr, seq, RLX, AGT);
        const int dst = (rank + 1 + ((lane < 7 ? lane : 0) + b % 7) % 7) & 7;
        unsigned* const fl = (unsigned*)(p.ws[dst] + OFF_P2F + (unsigned)b * 128u + (unsigned)rank * 4u);
        lds_wait_ge(aux + A_MS, 2u);
        if (lane < 7) __hip_atomic_store(fl, seq * 4u + 1u, RLX, SYS);
        VMCNT(0);
        int done = 0;
        while (done < 7) {
            lds_wait_ge(aux + A_P2, (unsigned)done + 1u);
            lds_wait_ge(aux + A_P2 + 1, (unsigned)done + 1u);
            const int d2 = (int)min(lds_rd(aux + A_P2), lds_rd(aux + A_P2 + 1));
            if (lane >= done && lane < d2) __hip_atomic_store(fl, seq * 4u + 2u, RLX, SYS);
            done = d2;
        }
        return;
    }

    const unsigned seq = __builtin_amdgcn_readfirstlane(seqv);
    if (lane == 0) __hip_atomic_store(ctr, seq, RLX, AGT);
    const u64 t0 = __builtin_amdgcn_s_memrealtime();
    VMCNT0_SEEN;
    START_BARRIER;

    if (wave == W_POLL) {
        const char* const pr = myws + OFF_PROG + (unsigned)(((idx ^ (idx >> 3)) & (PREP - 1)) * 32) * 128u;
        const char* const px = pr + (unsigned)tm * 128u;
        const char* const pw = pr + (unsigned)tn * 512u;
        const unsigned pbase = seq * 32u;
        unsigned sent = 0;
        bool stop = false;
        for (;;) {
            const u64 now = __builtin_amdgcn_s_memrealtime();
            u64 gap = 0;
            if (!stop) {
                int pv = 0, lo = NSUB;
                v8u fx, f0, f1, f2, f3;
                asm volatile("s_load_dwordx8 %0, %5, 0x0 glc\n\ts_load_dwordx8 %1, %6, 0x0 glc\n\ts_load_dwordx8 %2, %6, 0x80 glc\n\t"
                             "s_load_dwordx8 %3, %6, 0x100 glc\n\ts_load_dwordx8 %4, %6, 0x180 glc\n\ts_waitcnt lgkmcnt(0)"
                             : "=&s"(fx), "=&s"(f0), "=&s"(f1), "=&s"(f2), "=&s"(f3) : "s"(px), "s"(pw) : "memory");
#pragma unroll
                for (int c = 0; c < 8; ++c) {
                    const int m = min(min(min((int)(f0[c] - pbase), (int)(f1[c] - pbase)), min((int)(f2[c] - pbase), (int)(f3[c] - pbase))), (int)(fx[c] - pbase));
                    lo = min(lo, m);
                    pv = lane == c ? m : pv;
                }
                stop = lo >= NSUB;
                if (!stop && now - t0 > 50000ull) {
                    if (now - t0 > DEADLINE || __hip_atomic_load(tmo, RLX, AGT) == seq) {
                        __hip_atomic_store(tmo, seq, RLX, AGT);
                        pv = NSUB;
                        stop = true;
                        if (lane == 0) aux[A_DEAD] = 1u;
                    }
                }
                if (lane < 8) aux[A_PRG + lane] = (unsigned)pv;
                gap = lo >= NSUB - FAST_LEFT ? GAP_FAST : GAP_SLOW;
            }
            do {
                if (lds_rd(aux + A_CDONE) == 2u) { if (stop) return; }
                else {
                    const unsigned d = min(lds_rd(aux + A_SENT), lds_rd(aux + A_SENT + 1));
                    if (d >= sent + (d > FINE_FROM ? 1u : FSTEP)) { sent = d; if (lane < PREP) __hip_atomic_fetch_max(prog, pbase + d, RLX, SYS); }
                }
                __builtin_amdgcn_s_sleep(1);
            } while (__builtin_amdgcn_s_memrealtime() - now < gap);
        }
    }

    if (wave == W_SCHED) {
        const int csrc = (rank + lane) & 7;
        int nxt = lane < 8 ? 0 : NSUB;
        int asg = 0;
        while (asg < NUNIT) {
            const int prg = (int)aux[A_PRG + csrc];
            const v4u cs = *(const volatile LAS v4u*)(aux + A_CONS);
            const int lim = __builtin_amdgcn_readfirstlane(min(min(cs.x, cs.y), min(cs.z, cs.w))) + NSLOT;
            const int asg0 = asg;
            while (asg < lim) {
                const unsigned av = (unsigned)__ballot(prg > nxt);
                if (av == 0) break;
                const int c = __builtin_ctz(av);
                const int s = __builtin_amdgcn_readlane(nxt, c);
                if (lane == 0) aux[A_DESC + (asg & 15)] = (unsigned)(((rank + c) & 7) * NSUB + s) * UNIT;
                nxt = lane == c ? nxt + 1 : nxt;
                ++asg;
            }
            if (asg != asg0) { CFENCE; if (lane == 0) aux[A_ASG] = (unsigned)asg; }
            else __builtin_amdgcn_s_sleep(1);
        }
        lds_wait_ge(aux + A_CNT, 4u);
        if (lane == 0) __hip_atomic_store((u64*)(myws + OFF_WGMAX + (unsigned)b * 8u), ((u64)seq << 32) | aux[A_MAX], RLX, AGT);
        bool dead = __hip_atomic_load(tmo, RLX, AGT) == seq;
        if (b == 0) {
            auto rg = mk_rsrc(myws + OFF_WGMAX, NWG * 8);
            unsigned mx;
            for (;;) {
                v4u g0 = __builtin_amdgcn_raw_buffer_load_b128(rg, (unsigned)lane * 32u, 0, 17);
                v4u g1 = __builtin_amdgcn_raw_buffer_load_b128(rg, (unsigned)lane * 32u + 16u, 0, 17);
                unsigned tm_ = __hip_atomic_load(tmo, RLX, AGT);
                bool ok = g0.y == seq && g0.w == seq && g1.y == seq && g1.w == seq;
                mx = max(max(g0.x, g0.z), max(g1.x, g1.z));
                if (__all(ok)) break;
                if (tm_ == seq || expired(t0)) { __hip_atomic_store(tmo, seq, RLX, AGT); dead = true; break; }
            }
            mx = wave_max(mx);
            char* dst = p.ws[(lane >> 2) & 7] + OFF_RMAX + (unsigned)(lane & 3) * 512u + (unsigned)rank * 8u;
            if (lane < 32) __hip_atomic_store((u64*)dst, ((u64)seq << 32) | mx, RLX, SYS);
            if (lane == 0) aux[A_LEAD] = 1u;
            STAMP(7);
            const char* const ra = myws + OFF_RMAX;
            unsigned seen = 0, flip = 0;
            mx = 0;
            for (;;) {
                v16u g0, g1;
                asm volatile("s_load_dwordx16 %0, %2, 0x0 glc\n\ts_load_dwordx16 %1, %2, 0x200 glc\n\ts_waitcnt lgkmcnt(0)"
                             : "=&s"(g0), "=&s"(g1) : "s"(ra + flip) : "memory");
                flip ^= 1024u;
#pragma unroll
                for (int c = 0; c < 8; ++c) {
                    if (g0[2 * c + 1] == seq) { seen |= 1u << c; mx = max(mx, g0[2 * c]); }
                    if (g1[2 * c + 1] == seq) { seen |= 1u << c; mx = max(mx, g1[2 * c]); }
                }
                if (seen == 255u) break;
                const u64 now = __builtin_amdgcn_s_memrealtime();
                if (now - t0 > 50000ull) {
                    if (now - t0 > DEADLINE || __hip_atomic_load(tmo, RLX, AGT) == seq) { __hip_atomic_store(tmo, seq, RLX, AGT); dead = true; break; }
                }
            }
            if (lane < 32) __hip_atomic_store((u64*)(myws + OFF_GMAX + (unsigned)lane * 128u), ((u64)seq << 32) | mx, RLX, AGT);
            STAMP(14);
        }
        const char* const ga = myws + OFF_GMAX + (unsigned)(b & 31) * 128u;
        unsigned gm;
        for (;;) {
            u64 g;
            asm volatile("s_load_dwordx2 %0, %1, 0x0 glc\n\ts_waitcnt lgkmcnt(0)" : "=s"(g) : "s"(ga) : "memory");
            gm = (unsigned)g;
            if ((unsigned)(g >> 32) == seq) break;
            const u64 now = __builtin_amdgcn_s_memrealtime();
            if (now - t0 > 50000ull) {
                if (now - t0 > DEADLINE || __hip_atomic_load(tmo, RLX, AGT) == seq) { __hip_atomic_store(tmo, seq, RLX, AGT); dead = true; break; }
            }
            __builtin_amdgcn_s_sleep(1);
        }
        if (lane == 0) {
            aux[A_GM] = gm;
            if (dead) aux[A_DEAD] = 1u;
            CFENCE;
            aux[A_GO] = 1u;
            aux[A_TGO] = (unsigned)__builtin_amdgcn_s_memrealtime();
        }
        return;
    }

    {
        const int l = wave - 4;
        const char* const stage = myws + OFF_STAGE;
        const unsigned vx = (unsigned)tm * 4096u + (unsigned)lane * 16u;
        const unsigned vw = UNIT_X + (unsigned)tn * 8192u + (unsigned)lane * 16u;
        int slot = l;
        for (int n = l; n < NUNIT; n += NL) {
            lds_wait_ge(aux + A_ASG, (unsigned)(n + 1));
            const char* gb = stage + lds_rd(aux + A_DESC + (n & 15));
            const unsigned ld = ldsbase + (unsigned)slot * SLOT;
            glds4(gb, vx, ld);
            glds4(gb, vw, ld + 4096u);
            glds4(gb + 4096, vw, ld + 8192u);
            VMCNT(0);
            if (lane == 0) aux[A_FULL + slot] = (unsigned)(n + 1);
            slot += NL;
            if (slot >= NSLOT) slot -= NSLOT;
        }
        const int src = (rank + 1 + l) & 7;
        const char* const fl = myws + OFF_P2F + (unsigned)b * 128u + (unsigned)src * 4u;
        auto rp = mk_rsrc(myws + OFF_P2 + (unsigned)(src * NWG + b) * P2_PITCH, P2_PITCH);
        bool ok = wait_word(fl, seq * 4u + 1u, tmo, seq, t0);
        unsigned sel[8][6], off[8];
        {
            unsigned mk[4];
#pragma unroll
            for (int w = 0; w < 4; ++w) mk[w] = __builtin_amdgcn_raw_buffer_load_b32(rp, (unsigned)(w * 256 + lane * 4), 0, 17);
            unsigned base = 1024u;
#pragma unroll
            for (int w = 0; w < 4; ++w) {
                unsigned tot;
                const unsigned pre = lane_prefix((unsigned)__builtin_popcount(mk[w]), &tot);
                off[2 * w] = base + pre;
                off[2 * w + 1] = base + pre + (unsigned)__builtin_popcount(mk[w] & 0xffffu);
                base += tot;
#pragma unroll
                for (int mt = 0; mt < 2; ++mt) {
                    const unsigned m16 = (mk[w] >> (16 * mt)) & 0xffffu;
#pragma unroll
                    for (int t = 0; t < 4; ++t) {
                        const unsigned bits = (((m16 >> (4 * t)) & 15u) * 0x00204081u) & 0x01010101u;
                        const unsigned ix = bits * 0x01010100u + (unsigned)__builtin_popcount(m16 & ((1u << (4 * t)) - 1u)) * 0x01010101u;
                        if (t < 2) {
                            const unsigned ma = bits * 0xffu;
                            sel[2 * w + mt][t] = (ix & ma) | (0x0c0c0c0cu & ~ma);
                        } else {
                            const unsigned hi = (ix >> 3) & bits;
                            const unsigned ma = (bits ^ hi) * 0xffu, mb = hi * 0xffu;
                            sel[2 * w + mt][t] = (ix & ma) | (0x0c0c0c0cu & ~ma);
                            sel[2 * w + mt][t + 2] = (ix & 0x07070707u & mb) | (0x0c0c0c0cu & ~mb);
                        }
                    }
                }
            }
        }
        if (l == 0) STAMP(12);
        ok = ok && wait_word(fl, seq * 4u + 2u, tmo, seq, t0);
        if (l == 0) STAMP(13);
        v4u wv[8];
#pragma unroll
        for (int s = 0; s < 8; ++s) wv[s] = __builtin_amdgcn_raw_buffer_load_b128(rp, off[s], 0, 17);
        auto ro = mk_rsrc(p.out[rank], (unsigned)NCOL * 4096u);
        const unsigned ooff = (unsigned)((src >> 2) * 2048 + tm * 64 + fm) * NCOL + (unsigned)((src & 3) * 1024 + tn * 128 + fq * 16);
        const bool odd = fq & 1;
#pragma unroll
        for (int s = 0; s < 8; ++s) {
            const unsigned d0 = __builtin_amdgcn_perm(wv[s].y, wv[s].x, sel[s][0]);
            const unsigned d1 = __builtin_amdgcn_perm(wv[s].y, wv[s].x, sel[s][1]);
            const unsigned d2 = __builtin_amdgcn_perm(wv[s].y, wv[s].x, sel[s][2]) | __builtin_amdgcn_perm(wv[s].w, wv[s].z, sel[s][4]);
            const unsigned d3 = __builtin_amdgcn_perm(wv[s].y, wv[s].x, sel[s][3]) | __builtin_amdgcn_perm(wv[s].w, wv[s].z, sel[s][5]);
            v4u qv = {odd ? d1 : d0, odd ? d0 : d1, odd ? d3 : d2, odd ? d2 : d3};
            if (!ok) qv = v4u{0x80808080u, 0x80808080u, 0x80808080u, 0x80808080u};
            __builtin_amdgcn_raw_buffer_store_b128(qv, ro, ooff + (unsigned)(((s >> 2) * 32 + (s & 1) * 16) * NCOL + ((s >> 1) & 1) * 64), 0, OUT_AUX);
        }
        STAMP(16 + l);
    }
}

}

#define HIP_CHECK(expr) do { hipError_t _e = (expr); if (_e != hipSuccess) { \
    fprintf(stderr, "HIP error %s at %s:%d\n", hipGetErrorString(_e), __FILE__, __LINE__); \
    return; } } while (0)

extern "C" void kernel_launch(int rank, int nranks, ncclComm_t comm, hipStream_t stream,
                              void* const* d_in, const int* in_sizes, int n_in,
                              void* d_out, int out_size, const int* in_dtypes, int out_dtype,
                              void* const* d_out_all, void* const* d_ws_all, size_t ws_bytes) {
    (void)comm; (void)in_sizes; (void)n_in; (void)d_out; (void)in_dtypes; (void)out_dtype;
    if (nranks != NR || (unsigned)rank >= (unsigned)NR || ws_bytes < WS_NEED || out_size != NCOL * 4096) {
        fprintf(stderr, "unexpected problem geometry\n");
        return;
    }
    static bool attr_done[NR] = {false};
    if (!attr_done[rank]) {
        HIP_CHECK(hipFuncSetAttribute((const void*)fused_gemm_ar_q8, hipFuncAttributeMaxDynamicSharedMemorySize, LDS_BYTES));
        attr_done[rank] = true;
    }
    Params p;
    for (int r = 0; r < NR; ++r) { p.ws[r] = (char*)d_ws_all[r]; p.out[r] = (char*)d_out_all[r]; }
    p.x = (const char*)d_in[0];
    p.w = (const char*)d_in[1];
    p.rank = rank;
    hipLaunchKernelGGL(fused_gemm_ar_q8, dim3(NWG), dim3(NWAVE * 64), LDS_BYTES, stream, p);
    HIP_CHECK(hipGetLastError());
}
```

```cpp
#include <hip/hip_runtime.h>
#include <rccl/rccl.h>
#include <cstdio>
#include <cstdint>

#define COPY_D 3
#define PACE 342
#define GAP_SLOW 100
#define GAP_FAST 30
#define FAST_LEFT 3
#define OUT_AUX 16
#define PREP 2
#define VPOLL 0
#define GAP_VEC 15
#define VPACE 68
#define CMP_PRIO 2
#define MPACE 65
#define FSTEP 2u
#define FINE_FROM 12u
#define NSND 2
#define TGO_ADD 0u
#define MWAIT 400u
#define PSLEEP 2
#define CFLAG 0
#define LSLEEP 1
#define ILV 1
#define P2_D 3

namespace {

typedef short v4s __attribute__((ext_vector_type(4)));
typedef short v8s __attribute__((ext_vector_type(8)));
typedef __bf16 v8bf __attribute__((ext_vector_type(8)));
typedef float v4f __attribute__((ext_vector_type(4)));
typedef unsigned v4u __attribute__((ext_vector_type(4)));
typedef unsigned v3u __attribute__((ext_vector_type(3)));
typedef unsigned v8u __attribute__((ext_vector_type(8)));
typedef unsigned v16u __attribute__((ext_vector_type(16)));
typedef unsigned long long u64;
#define LAS __attribute__((address_space(3)))

constexpr int NR = 8;
constexpr int NCOL = 4096;
constexpr int KC = 512;
constexpr int NSUB = 16;
constexpr int NUNIT = NR * NSUB;
constexpr int NWG = 256;
constexpr int NL = 7;
constexpr int W_SCHED = 4 + NL, W_COPY = 5 + NL  , W_FETCH = 7 + NL, W_POLL = 8 + NL, NWAVE = 9 + NL;
constexpr unsigned UNIT_X = 2048u * 32u * 2u;
constexpr unsigned UNIT_W = 32u * 1024u * 2u;
constexpr unsigned UNIT = UNIT_X + UNIT_W;
constexpr unsigned OFF_CTR = 0;
constexpr unsigned OFF_TMO = 32768;
constexpr unsigned OFF_WGMAX = 40960;
constexpr unsigned OFF_RMAX = 45056;
constexpr unsigned OFF_GMAX = 49152;
constexpr unsigned OFF_DIAG = 65536;
constexpr unsigned OFF_PROG = 262144;
constexpr unsigned OFF_P2F = 294912;
constexpr unsigned OFF_STAGE = 1u << 20;
constexpr unsigned P2_PITCH = 24576;
constexpr unsigned OFF_P2 = OFF_STAGE + NR * NSUB * UNIT;
constexpr unsigned WS_NEED = OFF_P2 + NR * NWG * P2_PITCH;
constexpr unsigned SLOT = 12288;
#define NSLOT_ 12
#define CG_ 2
constexpr int NSLOT = NSLOT_;
constexpr unsigned LDS_AUX = NSLOT * SLOT;
constexpr unsigned LDS_CPY = LDS_AUX + 768;
constexpr int CG = CG_;
constexpr unsigned L_MASK = LDS_CPY + CG * 6144;
constexpr unsigned LDS_BYTES = L_MASK + 1024;
#define LSTAMP(n, k) do { } while (0)
constexpr unsigned L_STREAM = 0, L_PACK = 36864, L_DUMMY = 49152, L_DEC = 61440, DEC_PITCH = 8448;
constexpr int A_ASG = 0, A_CDONE = 1, A_SENT = 2, A_CONS = 4, A_MAX = 8, A_CNT = 9, A_CNT2 = 10, A_CNT3 = 11, A_GM = 12, A_DEAD = 13,
              A_GO = 14, A_CNT1 = 15, A_FULL = 32  , A_TOT = 44, A_FETCHED = 48, A_EXC = 49, A_TAKEN = 50, A_P2 = 52  , A_TGO = 16,
              A_LEAD = 17, A_PRG = 56, A_DESC = 64;
constexpr u64 DEADLINE = 100000000ull;

struct Params {
    char* ws[NR];
    char* out[NR];
    const char* x;
    const char* w;
    int rank;
};

#define SYS __HIP_MEMORY_SCOPE_SYSTEM
#define AGT __HIP_MEMORY_SCOPE_AGENT
#define WGS __HIP_MEMORY_SCOPE_WORKGROUP
#define RLX __ATOMIC_RELAXED
#define VMCNT(n) asm volatile("s_waitcnt vmcnt(" #n ")" ::: "memory")
#define VMCNTN(n) asm volatile("s_waitcnt vmcnt(%0)" :: "n"(n) : "memory")
#define VMCNT0_SEEN __builtin_amdgcn_s_waitcnt(0x0F70)
#define LGKM0 asm volatile("s_waitcnt lgkmcnt(0)" ::: "memory")
#define CFENCE asm volatile("" ::: "memory")
#define START_BARRIER do { __builtin_amdgcn_s_barrier(); CFENCE; } while (0)

#define STAMP(slot) do { } while (0)

__device__ __forceinline__ auto mk_rsrc(const void* base, unsigned bytes) {
    return __builtin_amdgcn_make_buffer_rsrc(const_cast<void*>(base), 0, bytes, 0x00020000);
}
__device__ __forceinline__ void glds4(const char* gbase, unsigned voff, unsigned lds) {
    unsigned keep;
    asm volatile("s_mov_b32 %0, m0\n\ts_mov_b32 m0, %3\n\ts_nop 0\n\t"
                 "global_load_lds_dwordx4 %1, %2 sc1\n\t"
                 "global_load_lds_dwordx4 %1, %2 offset:1024 sc1\n\t"
                 "global_load_lds_dwordx4 %1, %2 offset:2048 sc1\n\t"
                 "global_load_lds_dwordx4 %1, %2 offset:3072 sc1\n\t"
                 "s_mov_b32 m0, %0"
                 : "=&s"(keep) : "v"(voff), "s"(gbase), "s"(lds) : "memory");
}
__device__ __forceinline__ void glds1(const char* gbase, unsigned voff, unsigned lds) {
    unsigned keep;
    asm volatile("s_mov_b32 %0, m0\n\ts_mov_b32 m0, %3\n\ts_nop 0\n\t"
                 "global_load_lds_dwordx4 %1, %2\n\t"
                 "s_mov_b32 m0, %0"
                 : "=&s"(keep) : "v"(voff), "s"(gbase), "s"(lds) : "memory");
}
__device__ __forceinline__ bool expired(u64 t0) { return __builtin_amdgcn_s_memrealtime() - t0 > DEADLINE; }
__device__ __forceinline__ unsigned lds_rd(volatile LAS unsigned* a) { return __builtin_amdgcn_readfirstlane(*a); }
__device__ __forceinline__ void lds_wait_ge(volatile LAS unsigned* a, unsigned v) {
    while ((int)(lds_rd(a) - v) < 0) __builtin_amdgcn_s_sleep(1);
    CFENCE;
}
__device__ __forceinline__ void lds_spin_ge(volatile LAS unsigned* a, unsigned v) {
    while ((int)(lds_rd(a) - v) < 0) { }
    CFENCE;
}
__device__ __forceinline__ void lds_add(volatile LAS unsigned* a) {
    __hip_atomic_fetch_add((LAS unsigned*)a, 1u, RLX, WGS);
}
__device__ __forceinline__ unsigned wave_max(unsigned v) {
    v = max(v, (unsigned)__builtin_amdgcn_update_dpp(0, (int)v, 0x111, 0xf, 0xf, false));
    v = max(v, (unsigned)__builtin_amdgcn_update_dpp(0, (int)v, 0x112, 0xf, 0xf, false));
    v = max(v, (unsigned)__builtin_amdgcn_update_dpp(0, (int)v, 0x114, 0xf, 0xf, false));
    v = max(v, (unsigned)__builtin_amdgcn_update_dpp(0, (int)v, 0x118, 0xf, 0xf, false));
    return max(max((unsigned)__builtin_amdgcn_readlane((int)v, 15), (unsigned)__builtin_amdgcn_readlane((int)v, 31)),
               max((unsigned)__builtin_amdgcn_readlane((int)v, 47), (unsigned)__builtin_amdgcn_readlane((int)v, 63)));
}
__device__ __forceinline__ unsigned lane_prefix(unsigned c, unsigned* total) {
    unsigned pre = 0, tot = 0;
#pragma unroll
    for (int k = 0; k < 6; ++k) {
        const u64 m = __ballot((c >> k) & 1u);
        pre += __builtin_amdgcn_mbcnt_hi((unsigned)(m >> 32), __builtin_amdgcn_mbcnt_lo((unsigned)m, 0u)) << k;
        tot += (unsigned)__builtin_popcountll(m) << k;
    }
    *total = tot;
    return pre;
}
__device__ __forceinline__ unsigned quant4(v4f y, float inv) {
    unsigned r[4];
#pragma unroll
    for (int e = 0; e < 4; ++e) r[e] = __float_as_uint(fminf(__builtin_fmaf(y[e], inv, 12582912.0f), 12582912.0f + 127.0f));
    return __builtin_amdgcn_perm(r[1], r[0], 0x0c0c0400u) | __builtin_amdgcn_perm(r[3], r[2], 0x04000c0cu);
}
__device__ __forceinline__ unsigned wait_granule(const char* addr, unsigned* tmo, unsigned seq, u64 t0) {
    for (;;) {
        u64 f;
        asm volatile("s_load_dwordx2 %0, %1, 0x0 glc\n\ts_waitcnt lgkmcnt(0)" : "=s"(f) : "s"(addr) : "memory");
        if ((unsigned)(f >> 32) == seq) return (unsigned)f;
        const u64 now = __builtin_amdgcn_s_memrealtime();
        if (now - t0 > 50000ull) {
            if (now - t0 > DEADLINE || __hip_atomic_load(tmo, RLX, AGT) == seq) {
                __hip_atomic_store(tmo, seq, RLX, AGT);
                return 0xffffffffu;
            }
        }
        __builtin_amdgcn_s_sleep(PSLEEP);
    }
}

__global__ void __launch_bounds__(NWAVE * 64, 4) fused_gemm_ar_q8(Params p) {
    extern __shared__ __attribute__((aligned(16))) char smem[];
    const int tid = threadIdx.x;
    const int lane = tid & 63;
    const int wave = __builtin_amdgcn_readfirstlane(tid >> 6);
    const int b = blockIdx.x;
    const int rank = p.rank;
    char* const myws = p.ws[rank];
    unsigned* const tmo = (unsigned*)(myws + OFF_TMO);
    volatile LAS unsigned* aux = (volatile LAS unsigned*)(smem + LDS_AUX);
    const unsigned ldsbase = (unsigned)(size_t)(LAS char*)smem;
    const int xcd = b & 7, idx = b >> 3;
    const int tm = (xcd >> 1) * 8 + (idx & 7);
    const int tn = (xcd & 1) * 4 + (idx >> 3);
    const int fm = lane & 15, fq = lane >> 4;

    if (wave < 4) {
        if (tid < 64) aux[tid] = 0u;
        const int wm = wave >> 1, wn = wave & 1;
        v4f acc[2][4];
#pragma unroll
        for (int i = 0; i < 2; ++i)
#pragma unroll
            for (int j = 0; j < 4; ++j) acc[i][j] = v4f{0.f, 0.f, 0.f, 0.f};
        const unsigned xaddr = (unsigned)wm * 2048u + 16u * (unsigned)(4 * fm + (fq ^ ((4 - (fm >> 2)) & 3)));
        unsigned waddr;
        {
            const unsigned gg = (unsigned)fq, aa = (unsigned)((lane >> 2) & 3), bb = (unsigned)(lane & 3);
            const unsigned pp = bb >> 1, beta = bb & 1;
            const unsigned pos = ((2u * beta) ^ aa) | ((gg & 1u) << 2) | (pp << 3) | (aa << 4) | ((gg >> 1) << 6);
            waddr = 4096u + ((unsigned)wn * 2u + pp) * 2048u + 16u * pos + 8u * beta;
        }
        LGKM0;
        START_BARRIER;
        if (wave == 0) STAMP(0);
        {
            static_assert(NSLOT == 12 && NUNIT == 21 * 6 + 2, "ring unrolling");
            v8s xf[2][2], wf[2][4];
            unsigned xa = ldsbase + xaddr + 6u * SLOT, wa[8];
#pragma unroll
            for (int t = 0; t < 8; ++t)
                wa[t] = ldsbase + (waddr ^ (unsigned)(((t >> 2) & 1) << 4) ^ (unsigned)(((t >> 1) & 1) << 3) ^ (unsigned)((t & 1) << 7)) + 6u * SLOT;
            unsigned fa = ldsbase + LDS_AUX + (unsigned)(A_FULL + 6) * 4u;
            const unsigned consa = ldsbase + LDS_AUX + (unsigned)(A_CONS + wave) * 4u;
            auto rd = [&](int set, int i) {
#pragma unroll
                for (int t = 0; t < 4; ++t) {
                    v4s lo = __builtin_amdgcn_ds_read_tr16_b64_v4i16((LAS v4s*)(size_t)(wa[2 * t] + (unsigned)i * SLOT));
                    v4s hi = __builtin_amdgcn_ds_read_tr16_b64_v4i16((LAS v4s*)(size_t)(wa[2 * t + 1] + (unsigned)i * SLOT));
                    wf[set][t] = __builtin_shufflevector(lo, hi, 0, 1, 2, 3, 4, 5, 6, 7);
                    if (t == 0) {
#pragma unroll
                        for (int mt = 0; mt < 2; ++mt) xf[set][mt] = *(const LAS v8s*)(size_t)(xa + (unsigned)i * SLOT + mt * 1024);
                    }
                }
            };
            auto mm = [&](int set) {
#pragma unroll
                for (int t = 0; t < 4; ++t)
#pragma unroll
                    for (int mt = 0; mt < 2; ++mt)
                        acc[mt][t] = __builtin_amdgcn_mfma_f32_16x16x32_bf16((v8bf)wf[set][t], (v8bf)xf[set][mt], acc[mt][t], 0, 0, 0);
            };
            auto step = [&](int n, int i) {
                LGKM0;
                *(volatile LAS unsigned*)(size_t)consa = (unsigned)(n + 1);
                const unsigned f = *(volatile LAS unsigned*)(size_t)(fa + (unsigned)i * 4u);
                CFENCE;
                rd((i & 1) ^ 1, i);
                mm(i & 1);
                __builtin_amdgcn_sched_group_barrier(0x100, 2, 0); __builtin_amdgcn_sched_group_barrier(0x8, 1, 0);
                __builtin_amdgcn_sched_group_barrier(0x100, 2, 0); __builtin_amdgcn_sched_group_barrier(0x8, 1, 0);
                __builtin_amdgcn_sched_group_barrier(0x100, 2, 0); __builtin_amdgcn_sched_group_barrier(0x8, 1, 0);
                __builtin_amdgcn_sched_group_barrier(0x100, 2, 0); __builtin_amdgcn_sched_group_barrier(0x8, 1, 0);
                __builtin_amdgcn_sched_group_barrier(0x100, 2, 0); __builtin_amdgcn_sched_group_barrier(0x8, 4, 0);
                __builtin_amdgcn_sched_barrier(0);
                if ((int)(__builtin_amdgcn_readfirstlane(f) - (unsigned)(n + 2)) < 0) {
                    lds_wait_ge((volatile LAS unsigned*)(size_t)(fa + (unsigned)i * 4u), (unsigned)(n + 2));
                    rd((i & 1) ^ 1, i);
                }
            };
            __builtin_amdgcn_s_setprio(CMP_PRIO);
            lds_wait_ge(aux + A_FULL + 11, 1u);
            rd(0, 5);
            unsigned dl = 0u - 6u * SLOT, df = 0u - 24u;
            auto flip = [&]() {
                xa += dl;
#pragma unroll
                for (int t = 0; t < 8; ++t) wa[t] += dl;
                fa += df;
                dl = 0u - dl;
                df = 0u - df;
                asm volatile("" : "+v"(xa), "+v"(wa[0]), "+v"(wa[1]), "+v"(wa[2]), "+v"(wa[3]), "+v"(wa[4]), "+v"(wa[5]), "+v"(wa[6]), "+v"(wa[7]), "+v"(fa));
            };
            for (int n = 0; n < NUNIT - 2; n += 6) {
                flip();
#pragma unroll
                for (int i = 0; i < 6; ++i) step(n + i, i);
            }
            flip();
            step(NUNIT - 2, 0);
            mm(1);
        }
        if (wave == 0) STAMP(3);
        float mx = 0.f;
        unsigned m32 = 0;
#pragma unroll
        for (int i = 0; i < 2; ++i)
#pragma unroll
            for (int j = 0; j < 4; ++j)
#pragma unroll
                for (int e = 0; e < 4; ++e) {
                    acc[i][j][e] = fmaxf(acc[i][j][e], 0.f);
                    mx = fmaxf(mx, acc[i][j][e]);
                    m32 |= acc[i][j][e] > 0.f ? 1u << (i * 16 + j * 4 + e) : 0u;
                }
        *(LAS unsigned*)(smem + L_MASK + (unsigned)tid * 4u) = m32;
        const unsigned wmx = wave_max(__float_as_uint(mx));
        CFENCE;
        if (lane == 0) {
            __hip_atomic_fetch_max((LAS unsigned*)(aux + A_MAX), wmx, RLX, WGS);
            CFENCE;
            lds_add(aux + A_CNT);
        }
        unsigned tot;
        const unsigned pre = lane_prefix((unsigned)__builtin_popcount(m32), &tot);
        if (lane == 0) { aux[A_TOT + wave] = tot; CFENCE; lds_add(aux + A_CNT1); }
        lds_wait_ge(aux + A_CNT1, 4u);
        const v4u tt = *(const volatile LAS v4u*)(aux + A_TOT);
        const unsigned T = __builtin_amdgcn_readfirstlane(tt.x + tt.y + tt.z + tt.w);
        {
            unsigned a = L_STREAM + 4u * (pre + (wave > 0 ? tt.x : 0u) + (wave > 1 ? tt.y : 0u) + (wave > 2 ? tt.z : 0u));
            const unsigned dummy = L_DUMMY + (unsigned)tid * 4u;
#pragma unroll
            for (int i = 0; i < 2; ++i)
#pragma unroll
                for (int j = 0; j < 4; ++j)
#pragma unroll
                    for (int e = 0; e < 4; ++e) {
                        const bool on = acc[i][j][e] > 0.f;
                        *(LAS float*)(smem + (on ? a : dummy)) = acc[i][j][e];
                        a += on ? 4u : 0u;
                    }
        }
        CFENCE;
        if (lane == 0) lds_add(aux + A_CNT2);
        lds_wait_ge(aux + A_CNT2, 4u);
        v4f sv[2][4];
#pragma unroll
        for (int h = 0; h < 2; ++h)
#pragma unroll
            for (int j = 0; j < 4; ++j)
                sv[h][j] = *(const LAS v4f*)(smem + L_STREAM + (unsigned)((wave + 4 * h) * 4096 + lane * 64 + j * 16));
        lds_spin_ge(aux + A_GO, 1u);
        if (wave == 0) STAMP(4);
        const float amax = __uint_as_float(aux[A_GM]);
        const bool dead = aux[A_DEAD] != 0u;
        const float inv = amax > 0.f ? 127.0f / amax : 0.f;
        const v4u bad = {0x80808080u, 0x80808080u, 0x80808080u, 0x80808080u};
        const unsigned Vp = ((T + 15u) >> 4) * 12u;
#pragma unroll
        for (int h = 0; h < 2; ++h) {
            if (h == 0 || T > 4096u) {
                const unsigned c = (unsigned)((wave + 4 * h) * 64 + lane);
                const unsigned q0 = quant4(sv[h][0], inv), q1 = quant4(sv[h][1], inv), q2 = quant4(sv[h][2], inv), q3 = quant4(sv[h][3], inv);
                LAS unsigned* const dst = (LAS unsigned*)(smem + L_PACK + c * 12u);
                if (c * 16u < T) {
                    dst[0] = (q0 & 0x3f3f3f3fu) | ((q3 << 6) & 0xc0c0c0c0u);
                    dst[1] = (q1 & 0x3f3f3f3fu) | ((q3 << 4) & 0xc0c0c0c0u);
                    dst[2] = (q2 & 0x3f3f3f3fu) | ((q3 << 2) & 0xc0c0c0c0u);
                    if ((q0 | q1 | q2 | q3) & 0x40404040u) {
                        unsigned m = ((((q0 >> 6) & 0x01010101u) * 0x10204080u) >> 28) | (((((q1 >> 6) & 0x01010101u) * 0x10204080u) >> 28) << 4)
                                   | (((((q2 >> 6) & 0x01010101u) * 0x10204080u) >> 28) << 8) | (((((q3 >> 6) & 0x01010101u) * 0x10204080u) >> 28) << 12);
                        unsigned e = __hip_atomic_fetch_add((LAS unsigned*)(aux + A_EXC), (unsigned)__builtin_popcount(m), RLX, WGS);
                        while (m) {
                            *(LAS unsigned short*)(smem + L_PACK + Vp + 2u * e) = (unsigned short)(c * 16u + (unsigned)__builtin_ctz(m));
                            m &= m - 1u;
                            ++e;
                        }
                    }
                }
            }
        }
        CFENCE;
        if (lane == 0) lds_add(aux + A_CNT3);
        auto ro = mk_rsrc(p.out[rank], (unsigned)NCOL * 4096u);
        const unsigned ooff = (unsigned)((rank >> 2) * 2048 + tm * 64 + wm * 32 + fm) * NCOL + (unsigned)((rank & 3) * 1024 + tn * 128 + wn * 64 + fq * 16);
#pragma unroll
        for (int mt = 0; mt < 2; ++mt) {
            const unsigned d0 = quant4(acc[mt][0], inv), d1 = quant4(acc[mt][1], inv), d2 = quant4(acc[mt][2], inv), d3 = quant4(acc[mt][3], inv);
            const bool odd = fq & 1;
            v4u qv = {odd ? d1 : d0, odd ? d0 : d1, odd ? d3 : d2, odd ? d2 : d3};
            if (dead) qv = bad;
            __builtin_amdgcn_raw_buffer_store_b128(qv, ro, ooff + mt * 16 * NCOL, 0, OUT_AUX);
        }
        return;
    }

    unsigned* const ctr = (unsigned*)(myws + OFF_CTR) + (b * 16 + wave);
    const unsigned seqv = __hip_atomic_load(ctr, RLX, AGT) + 1u;
    const int q = b & 31;
    const int dest = (rank + (b >> 5)) & 7;

    unsigned* const prog = (unsigned*)(p.ws[dest] + OFF_PROG + (unsigned)((lane & (PREP - 1)) * 32 + q) * 128u + (unsigned)rank * 4u);

    const unsigned xo = (unsigned)((dest >> 2) * 2048 + q * 64 + (lane >> 2)) * (KC * 2) + (unsigned)((lane & 3) ^ ((4 - (lane >> 4)) & 3)) * 16u;
    const unsigned wo = (unsigned)((lane >> 4) | ((((lane >> 3) & 1) ^ (q & 1)) << 2) | (((lane >> 2) & 1) << 3)) * (NCOL * 2)
                      + (unsigned)((dest & 3) * 1024 + q * 32) * 2u + (unsigned)((lane & 3) ^ (lane >> 4)) * 16u;
    int st;
    unsigned seq;

    if (wave == W_COPY || wave == W_COPY + 1) {
        st = wave - W_COPY;
        __builtin_amdgcn_s_setprio(3);
        v4u d[3];
        d[0] = *(const v4u*)(p.x + xo + (unsigned)(st * 16 * (KC * 2)));
        d[1] = *(const v4u*)(p.x + xo + (unsigned)((st + 2) * 16 * (KC * 2)));
        d[2] = *(const v4u*)(p.w + wo + (unsigned)(st * 16 * (NCOL * 2)));
        if (st == 0) STAMP(8);
        auto rs = mk_rsrc(p.ws[dest] + OFF_STAGE + (size_t)rank * NSUB * UNIT, NSUB * UNIT);
        const unsigned ox = (unsigned)q * 4096u + (unsigned)(st * 1024 + lane * 16);
        const unsigned ow = UNIT_X + (unsigned)q * 2048u + (unsigned)(st * 1024 + lane * 16);
        const LAS char* const src = (const LAS char*)smem + LDS_CPY + (unsigned)(st * 1024 + lane * 16);
        const unsigned pace = dest != rank ? PACE : 0u;
        VMCNT0_SEEN;
        const unsigned tc = (unsigned)__builtin_amdgcn_s_memrealtime();
        int grp = 0;
        for (int s = 0; s < NSUB; ++s) {
            if (s > 0) {
                lds_wait_ge(aux + A_FETCHED, (unsigned)s);
#pragma unroll
                for (int j = 0; j < 3; ++j) d[j] = *(const LAS v4u*)(src + grp * 6144 + j * 2048);
                grp = grp == CG - 1 ? 0 : grp + 1;
                LGKM0;
                if (lane == 0) aux[A_TAKEN + st] = (unsigned)s;
            }
#pragma unroll
            for (int j = 0; j < 3; ++j) {
                VMCNT(1);
                if (j == 1 && s > 0) {
                    if (lane == 0) aux[A_SENT + st] = (unsigned)s;
                    if (CFLAG && s > (int)FINE_FROM) {
                        CFENCE;
                        if (lds_rd(aux + A_SENT + (st ^ 1)) >= (unsigned)s && lane < PREP)
                            __hip_atomic_fetch_max(prog, seqv * 32u + (unsigned)s, RLX, SYS);
                    }
                }
                const unsigned due = (unsigned)(s * 6 + j * 2 + st) * pace / 6u;
                while ((unsigned)__builtin_amdgcn_s_memrealtime() - tc < due) __builtin_amdgcn_s_sleep(1);
                __builtin_amdgcn_raw_buffer_store_b128(d[j], rs, (j < 2 ? ox + j * 2048 : ow) + (unsigned)s * UNIT, 0, 17);
                if (s == 0 && j == 0) { if (st == 0) STAMP(9); START_BARRIER; }
            }
        }
        VMCNT(0);
        seq = __builtin_amdgcn_readfirstlane(seqv);
        unsigned old = 0;
        if (lane == 0) old = __hip_atomic_fetch_add((LAS unsigned*)(aux + A_CDONE), 1u, RLX, WGS);
        if (__builtin_amdgcn_readfirstlane(old) == 1u && lane < PREP) __hip_atomic_fetch_max(prog, seq * 32u + (unsigned)NSUB, RLX, SYS);
        if (lane == 0) __hip_atomic_store(ctr, seq, RLX, AGT);
        STAMP(10);
    } else {

    if (wave == W_FETCH) {
        auto fetch = [&](int g) {
            const unsigned l0 = ldsbase + LDS_CPY + (unsigned)(g % CG) * 6144u;
#pragma unroll
            for (int j = 0; j < 4; ++j) glds1(p.x, xo + (unsigned)((g + 1) * 64 + j * 16 * (KC * 2)), l0 + j * 1024u);
#pragma unroll
            for (int j = 0; j < 2; ++j) glds1(p.w, wo + (unsigned)(((g + 1) * 32 + j * 16) * (NCOL * 2)), l0 + 4096u + j * 1024u);
        };
        constexpr int NG = NSUB - 1;
#pragma unroll
        for (int g = 0; g < CG; ++g) fetch(g);
        START_BARRIER;
#pragma unroll
        for (int g = 0; g < NG; ++g) {
            VMCNTN(6 * ((CG + g < NG ? CG + g : NG) - g - 1));
            if (lane == 0) aux[A_FETCHED] = (unsigned)(g + 1);
            if (g + CG < NG) { lds_wait_ge(aux + A_TAKEN, (unsigned)(g + 1)); lds_wait_ge(aux + A_TAKEN + 1, (unsigned)(g + 1)); fetch(g + CG); }
        }
        VMCNT0_SEEN;
        seq = __builtin_amdgcn_readfirstlane(seqv);
        if (lane == 0) __hip_atomic_store(ctr, seq, RLX, AGT);
        const int dst = (rank + 1 + ((lane < 7 ? lane : 0) + b % 7) % 7) & 7;
        u64* const fl = (u64*)(p.ws[dst] + OFF_P2F + (unsigned)b * 128u + (unsigned)rank * 8u);
        lds_wait_ge(aux + A_CNT3, 4u);
        const u64 fv = ((u64)seq << 32) | (aux[A_DEAD] ? 0xffffffffu : aux[A_EXC]);
        int done = 0;
        while (done < 7) {
            int d2;
            for (;;) {
                const v4u pp = *(const volatile LAS v4u*)(aux + A_P2);
                d2 = (int)__builtin_amdgcn_readfirstlane(NSND == 3 ? min(min(pp.x, pp.y), pp.z) : min(pp.x, pp.y));
                if (d2 > done) break;
                if (done < 6) __builtin_amdgcn_s_sleep(1);
            }
            if (lane >= done && lane < d2) __hip_atomic_store(fl, fv, RLX, SYS);
            done = d2;
        }
        return;
    }

    seq = __builtin_amdgcn_readfirstlane(seqv);
    if (lane == 0) __hip_atomic_store(ctr, seq, RLX, AGT);
    const u64 t0 = __builtin_amdgcn_s_memrealtime();
    VMCNT0_SEEN;
    START_BARRIER;

    if (wave == W_POLL) {
        const char* px = myws + OFF_PROG + (unsigned)tm * 128u;
        const char* pw = myws + OFF_PROG + (unsigned)tn * 512u;
        unsigned radd = (unsigned)(((idx ^ (idx >> 3)) & (PREP - 1)) * 32) * 128u;
        int pvk = 0;
        const unsigned pbase = seq * 32u;
        unsigned sent = 0;
        bool stop = false, fin = false, fast = false;
        while (!fin) {
            const u64 now = __builtin_amdgcn_s_memrealtime();
            u64 gap = 0;
            if (!stop) {
                int pv = 0, lo = NSUB;
                v8u fx, f0, f1, f2, f3;
                asm volatile("s_load_dwordx8 %0, %5, 0x0 glc\n\ts_load_dwordx8 %1, %6, 0x0 glc\n\ts_load_dwordx8 %2, %6, 0x80 glc\n\t"
                             "s_load_dwordx8 %3, %6, 0x100 glc\n\ts_load_dwordx8 %4, %6, 0x180 glc\n\ts_waitcnt lgkmcnt(0)"
                             : "=&s"(fx), "=&s"(f0), "=&s"(f1), "=&s"(f2), "=&s"(f3) : "s"(px), "s"(pw) : "memory");
#pragma unroll
                for (int c = 0; c < 8; ++c) {
                    const int m = min(min(min((int)(f0[c] - pbase), (int)(f1[c] - pbase)), min((int)(f2[c] - pbase), (int)(f3[c] - pbase))), (int)(fx[c] - pbase));
                    lo = min(lo, m);
                    pv = lane == c ? m : pv;
                }
                pv = pvk = max(pv, pvk);
                if (lo >= (int)FINE_FROM) { px += radd; pw += radd; radd = 0; fast = true; }
                stop = lo >= NSUB;
                if (!stop && now - t0 > 50000ull) {
                    if (now - t0 > DEADLINE || __hip_atomic_load(tmo, RLX, AGT) == seq) {
                        __hip_atomic_store(tmo, seq, RLX, AGT);
                        pv = NSUB;
                        stop = true;
                        if (lane == 0) aux[A_DEAD] = 1u;
                    }
                }
                if (lane < 8) aux[A_PRG + lane] = (unsigned)pv;
                if (stop) STAMP(15);
                gap = fast ? GAP_FAST : GAP_SLOW;
            }
            do {
                if (lds_rd(aux + A_CDONE) == 2u) { if (stop) { fin = true; break; } }
                else {
                    const unsigned d = min(lds_rd(aux + A_SENT), lds_rd(aux + A_SENT + 1));
                    if (d >= sent + (d > FINE_FROM ? 1u : FSTEP) && !(CFLAG && d > FINE_FROM)) { sent = d; if (lane < (d > FINE_FROM ? PREP : 1)) __hip_atomic_fetch_max(prog, pbase + d, RLX, SYS); }
                }
                __builtin_amdgcn_s_sleep(1);
            } while (__builtin_amdgcn_s_memrealtime() - now < gap);
        }
        if (NSND < 3) return;
        st = 2;
        __builtin_amdgcn_s_setprio(3);
    } else if (wave == W_SCHED) {
        const int csrc = (rank + lane) & 7;
        int nxt = lane < 8 ? 0 : NSUB;
        int asg = 0;
        while (asg < NUNIT) {
            const int prg = (int)aux[A_PRG + csrc];
            unsigned av = (unsigned)__ballot(prg > nxt);
            if (av == 0) { __builtin_amdgcn_s_sleep(1); continue; }
            while (av) {
                const int k = __builtin_popcount(av & ((1u << (lane & 7)) - 1u));
                if (prg > nxt) { aux[A_DESC + asg + k] = (unsigned)(csrc * NSUB + nxt) * UNIT; ++nxt; }
                asg += __builtin_popcount(av);
                CFENCE;
                if (lane == 0) aux[A_ASG] = (unsigned)asg;
                av = (unsigned)__ballot(prg > nxt);
            }
        }
        STAMP(23);
        lds_wait_ge(aux + A_CNT, 4u);
        if (lane == 0) __hip_atomic_store((u64*)(myws + OFF_WGMAX + (unsigned)b * 8u), ((u64)seq << 32) | aux[A_MAX], RLX, AGT);
        bool dead = __hip_atomic_load(tmo, RLX, AGT) == seq;
        if (b == 0) {
            auto rg = mk_rsrc(myws + OFF_WGMAX, NWG * 8);
            unsigned mx;
            for (;;) {
                v4u g0 = __builtin_amdgcn_raw_buffer_load_b128(rg, (unsigned)lane * 32u, 0, 17);
                v4u g1 = __builtin_amdgcn_raw_buffer_load_b128(rg, (unsigned)lane * 32u + 16u, 0, 17);
                unsigned tm_ = __hip_atomic_load(tmo, RLX, AGT);
                bool ok = g0.y == seq && g0.w == seq && g1.y == seq && g1.w == seq;
                mx = max(max(g0.x, g0.z), max(g1.x, g1.z));
                if (__all(ok)) break;
                if (tm_ == seq || expired(t0)) { __hip_atomic_store(tmo, seq, RLX, AGT); dead = true; break; }
            }
            mx = wave_max(mx);
            char* dst = p.ws[(lane >> 2) & 7] + OFF_RMAX + (unsigned)(lane & 3) * 512u + (unsigned)rank * 8u;
            if (lane < 32) __hip_atomic_store((u64*)dst, ((u64)seq << 32) | mx, RLX, SYS);
            if (lane == 0) aux[A_LEAD] = 1u;
            STAMP(7);
            const char* const ra = myws + OFF_RMAX;
            unsigned seen = 0, flip = 0;
            mx = 0;
            for (;;) {
                v16u g0, g1;
                asm volatile("s_load_dwordx16 %0, %2, 0x0 glc\n\ts_load_dwordx16 %1, %2, 0x200 glc\n\ts_waitcnt lgkmcnt(0)"
                             : "=&s"(g0), "=&s"(g1) : "s"(ra + flip) : "memory");
                flip ^= 1024u;
#pragma unroll
                for (int c = 0; c < 8; ++c) {
                    if (g0[2 * c + 1] == seq) { seen |= 1u << c; mx = max(mx, g0[2 * c]); }
                    if (g1[2 * c + 1] == seq) { seen |= 1u << c; mx = max(mx, g1[2 * c]); }
                }
                if (seen == 255u) break;
                const u64 now = __builtin_amdgcn_s_memrealtime();
                if (now - t0 > 50000ull) {
                    if (now - t0 > DEADLINE || __hip_atomic_load(tmo, RLX, AGT) == seq) { __hip_atomic_store(tmo, seq, RLX, AGT); dead = true; break; }
                }
            }
            if (lane < 32) __builtin_amdgcn_raw_buffer_store_b128(v4u{mx, seq, (unsigned)__builtin_amdgcn_s_memrealtime(), seq}, mk_rsrc(myws + OFF_GMAX, 4096u), (unsigned)lane * 128u, 0, 16);
            STAMP(14);
        }
        const char* const ga = myws + OFF_GMAX + (unsigned)(b & 31) * 128u;
        unsigned gm, tg;
        for (;;) {
            v4u g;
            asm volatile("s_load_dwordx4 %0, %1, 0x0 glc\n\ts_waitcnt lgkmcnt(0)" : "=s"(g) : "s"(ga) : "memory");
            gm = g.x;
            tg = g.z;
            if (g.y == seq && g.w == seq) break;
            const u64 now = __builtin_amdgcn_s_memrealtime();
            if (now - t0 > 50000ull) {
                if (now - t0 > DEADLINE || __hip_atomic_load(tmo, RLX, AGT) == seq) { __hip_atomic_store(tmo, seq, RLX, AGT); dead = true; break; }
            }
        }
        if (dead) tg = (unsigned)__builtin_amdgcn_s_memrealtime();
        if (lane == 0) {
            aux[A_GM] = gm;
            aux[A_TGO] = tg + TGO_ADD;
            if (dead) aux[A_DEAD] = 1u;
            CFENCE;
            aux[A_GO] = 1u;
        }
        return;
    } else {
        const int l = wave - 4;
        const char* const stage = myws + OFF_STAGE;
        const unsigned vx = (unsigned)tm * 4096u + (unsigned)lane * 16u;
        const unsigned vw = UNIT_X + (unsigned)tn * 8192u + (unsigned)lane * 16u;
        auto issue = [&](int n, int sl) {
            const char* gb = stage + lds_rd(aux + A_DESC + n);
            const unsigned ld = ldsbase + (unsigned)sl * SLOT;
            LSTAMP(n, 1);
            glds4(gb, vx, ld);
            glds4(gb, vw, ld + 4096u);
            glds4(gb + 4096, vw, ld + 8192u);
        };
        auto ready = [&](int n) {
            const unsigned as = aux[A_ASG];
            const v4u cs = *(const volatile LAS v4u*)(aux + A_CONS);
            const bool r = __builtin_amdgcn_readfirstlane(min(as, min(min(cs.x, cs.y), min(cs.z, cs.w)) + NSLOT)) > (unsigned)n;
            CFENCE;
            return r;
        };
        int slot = l == 0 ? NSLOT - 1 : l - 1;
        while (!ready(l)) __builtin_amdgcn_s_sleep(LSLEEP);
        issue(l, slot);
        for (int n = l;;) {
            const int nn = n + NL, ns = slot + NL >= NSLOT ? slot + NL - NSLOT : slot + NL;
            const bool next = nn < NUNIT && ready(nn);
            if (next) { issue(nn, ns); VMCNT(12); } else VMCNT(0);
            if (lane == 0) aux[A_FULL + slot] = (unsigned)(n + 1);
            LSTAMP(n, 2);
            if (nn >= NUNIT) break;
            if (!next) { while (!ready(nn)) __builtin_amdgcn_s_sleep(LSLEEP); issue(nn, ns); }
            n = nn;
            slot = ns;
        }
        const int src = (rank + 1 + l) & 7;
        const char* const fl = myws + OFF_P2F + (unsigned)b * 128u + (unsigned)src * 8u;
        auto rp = mk_rsrc(myws + OFF_P2 + (unsigned)(src * NWG + b) * P2_PITCH, P2_PITCH);
        lds_wait_ge(aux + A_GO, 1u);
        {
            const unsigned tg = lds_rd(aux + A_TGO);
            while ((unsigned)__builtin_amdgcn_s_memrealtime() - tg < MWAIT) __builtin_amdgcn_s_sleep(2);
        }
        unsigned mk[4];
#pragma unroll
        for (int w = 0; w < 4; ++w) mk[w] = __builtin_amdgcn_raw_buffer_load_b32(rp, (unsigned)(w * 256 + lane * 4), 0, 17);
        unsigned nexc = 0;
        unsigned sel[8][6], off[8];
        v3u pk[8];
        unsigned ex0, xi;
        int nit;
        for (int pass = 0;; ++pass) {
            unsigned base = 0;
#pragma unroll
            for (int w = 0; w < 4; ++w) {
                unsigned tot;
                const unsigned pre = lane_prefix((unsigned)__builtin_popcount(mk[w]), &tot);
                off[2 * w] = base + pre;
                off[2 * w + 1] = base + pre + (unsigned)__builtin_popcount(mk[w] & 0xffffu);
                base += tot;
#pragma unroll
                for (int mt = 0; mt < 2; ++mt) {
                    const unsigned m16 = (mk[w] >> (16 * mt)) & 0xffffu;
#pragma unroll
                    for (int t = 0; t < 4; ++t) {
                        const unsigned bits = (((m16 >> (4 * t)) & 15u) * 0x00204081u) & 0x01010101u;
                        const unsigned ix = bits * 0x01010100u + (unsigned)__builtin_popcount(m16 & ((1u << (4 * t)) - 1u)) * 0x01010101u;
                        if (t < 2) {
                            const unsigned ma = bits * 0xffu;
                            sel[2 * w + mt][t] = (ix & ma) | (0x0c0c0c0cu & ~ma);
                        } else {
                            const unsigned hi = (ix >> 3) & bits;
                            const unsigned ma = (bits ^ hi) * 0xffu, mb = hi * 0xffu;
                            sel[2 * w + mt][t] = (ix & ma) | (0x0c0c0c0cu & ~ma);
                            sel[2 * w + mt][t + 2] = (ix & 0x07070707u & mb) | (0x0c0c0c0cu & ~mb);
                        }
                    }
                }
            }
#pragma unroll
            for (int i = 0; i < 8; ++i)
                asm volatile("" : "+v"(sel[i][0]), "+v"(sel[i][1]), "+v"(sel[i][2]), "+v"(sel[i][3]), "+v"(sel[i][4]), "+v"(sel[i][5]), "+v"(off[i]));
            if (pass == 0) {
                if (l == 0) STAMP(12);
                nexc = wait_granule(fl, tmo, seq, t0);
                if (l == 0) STAMP(13);
            }
            nit = (int)((base + 1023u) >> 10);
#pragma unroll
            for (int i = 0; i < 8; ++i)
                if (i < nit) pk[i] = __builtin_amdgcn_raw_buffer_load_b96(rp, 1024u + (unsigned)(i * 64 + lane) * 12u, 0, 17);
            ex0 = 1024u + ((base + 15u) >> 4) * 12u;
            xi = __builtin_amdgcn_raw_buffer_load_b16(rp, ex0 + 2u * (unsigned)lane, 0, 17);
            if (pass > 0) break;
            bool same = true;
#pragma unroll
            for (int w = 0; w < 4; ++w) {
                const unsigned m2 = __builtin_amdgcn_raw_buffer_load_b32(rp, (unsigned)(w * 256 + lane * 4), 0, 17);
                same = same && m2 == mk[w];
                mk[w] = m2;
            }
            if (__all(same)) break;
        }
        const bool ok = nexc != 0xffffffffu;
        LAS char* const dec = (LAS char*)smem + L_DEC + (unsigned)l * DEC_PITCH;
        {
#pragma unroll
            for (int i = 0; i < 8; ++i) {
                if (i < nit) {
                    const v4u q = {pk[i].x & 0x3f3f3f3fu, pk[i].y & 0x3f3f3f3fu, pk[i].z & 0x3f3f3f3fu,
                                   ((pk[i].x >> 6) & 0x03030303u) | ((pk[i].y >> 4) & 0x0c0c0c0cu) | ((pk[i].z >> 2) & 0x30303030u)};
                    *(LAS v4u*)(dec + (unsigned)(i * 1024 + lane * 16)) = q;
                }
            }
            CFENCE;
            for (unsigned e = (unsigned)lane; ok && e < min(nexc, 8192u); e += 64u) {
                if (e >= 64u) xi = __builtin_amdgcn_raw_buffer_load_b16(rp, ex0 + 2u * e, 0, 17);
                __hip_atomic_fetch_or((LAS unsigned*)(dec + (xi & 0x1ffcu)), 0x40u << (8u * (xi & 3u)), RLX, WGS);
            }
            CFENCE;
        }
        auto ro = mk_rsrc(p.out[rank], (unsigned)NCOL * 4096u);
        const unsigned ooff = (unsigned)((src >> 2) * 2048 + tm * 64 + fm) * NCOL + (unsigned)((src & 3) * 1024 + tn * 128 + fq * 16);
        const bool odd = fq & 1;
#pragma unroll
        for (int s = 0; s < 8; ++s) {
            const LAS unsigned* const wp = (const LAS unsigned*)(dec + (off[s] & ~3u));
            const unsigned r0 = wp[0], r1 = wp[1], r2 = wp[2], r3 = wp[3], r4 = wp[4];
            const unsigned w0 = __builtin_amdgcn_alignbyte(r1, r0, off[s]), w1 = __builtin_amdgcn_alignbyte(r2, r1, off[s]);
            const unsigned w2 = __builtin_amdgcn_alignbyte(r3, r2, off[s]), w3 = __builtin_amdgcn_alignbyte(r4, r3, off[s]);
            const unsigned d0 = __builtin_amdgcn_perm(w1, w0, sel[s][0]);
            const unsigned d1 = __builtin_amdgcn_perm(w1, w0, sel[s][1]);
            const unsigned d2 = __builtin_amdgcn_perm(w1, w0, sel[s][2]) | __builtin_amdgcn_perm(w3, w2, sel[s][4]);
            const unsigned d3 = __builtin_amdgcn_perm(w1, w0, sel[s][3]) | __builtin_amdgcn_perm(w3, w2, sel[s][5]);
            v4u qv = {odd ? d1 : d0, odd ? d0 : d1, odd ? d3 : d2, odd ? d2 : d3};
            if (!ok) qv = v4u{0x80808080u, 0x80808080u, 0x80808080u, 0x80808080u};
            __builtin_amdgcn_raw_buffer_store_b128(qv, ro, ooff + (unsigned)(((s >> 2) * 32 + (s & 1) * 16) * NCOL + ((s >> 1) & 1) * 64), 0, OUT_AUX);
        }
        STAMP(16 + l);
        return;
    }
    }

    const unsigned poff = OFF_P2 + (unsigned)(rank * NWG + b) * P2_PITCH;
    const int rot = b % 7;
    lds_wait_ge(aux + A_CNT, 4u);
    const unsigned tm0 = (unsigned)__builtin_amdgcn_s_memrealtime();
    {
        const v4u mk = *(const LAS v4u*)(smem + L_MASK + (unsigned)lane * 16u);
        if (b == 0) lds_wait_ge(aux + A_LEAD, 1u);
        for (int k = st; k < 7; k += NSND) {
            const int dst = (rank + 1 + (k + rot) % 7) & 7;
            VMCNT(1);
            if (b != 0) while ((unsigned)__builtin_amdgcn_s_memrealtime() - tm0 < (unsigned)k * MPACE) __builtin_amdgcn_s_sleep(1);
            __builtin_amdgcn_raw_buffer_store_b128(mk, mk_rsrc(p.ws[dst] + poff, 1024u), (unsigned)lane * 16u, 0, 17);
        }
    }
    if (st == 0) STAMP(11);
    int last = 0, pub = 0;
    lds_spin_ge(aux + A_CNT3, 4u);
    CFENCE;
    const v4u tt = *(const volatile LAS v4u*)(aux + A_TOT);
    const unsigned Tr = (__builtin_amdgcn_readfirstlane(((tt.x + tt.y + tt.z + tt.w + 15u) >> 4) * 12u + 2u * aux[A_EXC]) + 63u) & ~63u;
    const int np = (int)((Tr + 1023u) >> 10);
    const unsigned ps = np ? (((Tr >> 6) + (unsigned)np - 1u) / (unsigned)np) << 6 : 0u;
    unsigned tv = lds_rd(aux + A_TGO);
    if ((int)(tm0 + 7u * MPACE - tv) > 0) tv = tm0 + 7u * MPACE;
    for (int k = 0, j = st; ; j += NSND) {
        while (j >= np && k < 7) { j -= np; ++k; }
        if (k >= 7) break;
        const unsigned o = (unsigned)j * ps;
        const v4u pv = *(const LAS v4u*)(smem + L_PACK + o + (unsigned)lane * 16u);
        char* const dp = p.ws[(rank + 1 + (k + rot) % 7) & 7] + poff + 1024u + o;
        VMCNT(1);
        if (last > pub) {
            if (lane == 0) aux[A_P2 + st] = (unsigned)last;
            pub = last;
        }
        const unsigned due = (((unsigned)k * Tr + o) * VPACE) >> 10;
        while ((int)((unsigned)__builtin_amdgcn_s_memrealtime() - tv - due) < 0) __builtin_amdgcn_s_sleep(1);
        __builtin_amdgcn_raw_buffer_store_b128(pv, mk_rsrc(dp, o < Tr ? min(ps, Tr - o) : 0u), (unsigned)lane * 16u, 0, 17);
        last = k;
    }
    VMCNT(0);
    if (lane == 0) aux[A_P2 + st] = 7u;
    STAMP(5);
}

}

#define HIP_CHECK(expr) do { hipError_t _e = (expr); if (_e != hipSuccess) { \
    fprintf(stderr, "HIP error %s at %s:%d\n", hipGetErrorString(_e), __FILE__, __LINE__); \
    return; } } while (0)

extern "C" void kernel_launch(int rank, int nranks, ncclComm_t comm, hipStream_t stream,
                              void* const* d_in, const int* in_sizes, int n_in,
                              void* d_out, int out_size, const int* in_dtypes, int out_dtype,
                              void* const* d_out_all, void* const* d_ws_all, size_t ws_bytes) {
    (void)comm; (void)in_sizes; (void)n_in; (void)d_out; (void)in_dtypes; (void)out_dtype;
    if (nranks != NR || (unsigned)rank >= (unsigned)NR || ws_bytes < WS_NEED || out_size != NCOL * 4096) {
        fprintf(stderr, "unexpected problem geometry\n");
        return;
    }
    static bool attr_done[NR] = {false};
    if (!attr_done[rank]) {
        HIP_CHECK(hipFuncSetAttribute((const void*)fused_gemm_ar_q8, hipFuncAttributeMaxDynamicSharedMemorySize, LDS_BYTES));
        attr_done[rank] = true;
    }
    Params p;
    for (int r = 0; r < NR; ++r) { p.ws[r] = (char*)d_ws_all[r]; p.out[r] = (char*)d_out_all[r]; }
    p.x = (const char*)d_in[0];
    p.w = (const char*)d_in[1];
    p.rank = rank;
    hipLaunchKernelGGL(fused_gemm_ar_q8, dim3(NWG), dim3(NWAVE * 64), LDS_BYTES, stream, p);
    HIP_CHECK(hipGetLastError());
}
```

```cpp
#include <hip/hip_runtime.h>
#include <rccl/rccl.h>
#include <cstdio>
#include <cstdint>

#define PACE 342u
#define MPACE 66u
#define VPACE 66u
#define MWAIT 400u
#define GAP_SLOW 100u
#define GAP_FAST 30u

#define XADV 80u

namespace {

typedef short v4s __attribute__((ext_vector_type(4)));
typedef short v8s __attribute__((ext_vector_type(8)));
typedef __bf16 v8bf __attribute__((ext_vector_type(8)));
typedef float v4f __attribute__((ext_vector_type(4)));
typedef unsigned v4u __attribute__((ext_vector_type(4)));
typedef unsigned v3u __attribute__((ext_vector_type(3)));
typedef unsigned v8u __attribute__((ext_vector_type(8)));
typedef unsigned v16u __attribute__((ext_vector_type(16)));
typedef unsigned long long u64;
#define LAS __attribute__((address_space(3)))

constexpr int NR = 8;
constexpr int NCOL = 4096;
constexpr int KC = 512;
constexpr int NSUB = 16;
constexpr int NUNIT = NR * NSUB;
constexpr int NWG = 256;
constexpr int NL = 7;
constexpr int W_SCHED = 4 + NL, W_COPY = 5 + NL  , W_FETCH = 7 + NL, W_POLL = 8 + NL, NWAVE = 9 + NL;
constexpr int FREP = 4, PREP = 2;
constexpr unsigned FINE_FROM = 12, FSTEP = 2;
constexpr unsigned UNIT_X = 2048u * 32u * 2u;
constexpr unsigned UNIT_W = 32u * 1024u * 2u;
constexpr unsigned UNIT = UNIT_X + UNIT_W;
constexpr unsigned OFF_CTR = 0;
constexpr unsigned OFF_TMO = 32768;
constexpr unsigned OFF_WGMAX = 40960;
constexpr unsigned OFF_RMAX = 45056;
constexpr unsigned OFF_GMAX = 49152;
constexpr unsigned OFF_DIAG = 65536;
constexpr unsigned OFF_PROG = 262144;
constexpr unsigned OFF_P2F = 327680;
constexpr unsigned OFF_STAGE = 1u << 20;
constexpr unsigned P2_PITCH = 10240;
constexpr unsigned OFF_P2 = OFF_STAGE + NR * NSUB * UNIT;
constexpr unsigned WS_NEED = OFF_P2 + NR * NWG * P2_PITCH;
constexpr unsigned SLOT = 12288;
constexpr int NSLOT = 12;
constexpr unsigned LDS_AUX = NSLOT * SLOT;
constexpr unsigned LDS_CPY = LDS_AUX + 768;
constexpr int CG = 2;
constexpr unsigned L_MASK = LDS_CPY + CG * 6144;
constexpr unsigned LDS_BYTES = L_MASK + 1024;
constexpr unsigned L_STREAM = 0, L_PACK = 36864, L_LIST = 45056, L_PLANE = 47104, L_DUMMY = 49152, L_DEC = 61440, DEC_PITCH = 8448;
constexpr int A_ASG = 0, A_CDONE = 1, A_SENT = 2, A_CONS = 4, A_MAX = 8, A_CNT = 9, A_CNT2 = 10, A_CNT3 = 11, A_GM = 12, A_DEAD = 13,
              A_GO = 14, A_CNT1 = 15, A_FULL = 32  , A_TOT = 44, A_FETCHED = 48, A_EXC = 49, A_TAKEN = 50, A_P2 = 52  , A_TGO = 16,
              A_LEAD = 17, A_CNTM = 18, A_PDONE = 19, A_PRG = 56, A_DESC = 64;
constexpr u64 DEADLINE = 100000000ull;

struct Params {
    char* ws[NR];
    char* out[NR];
    const char* x;
    const char* w;
    int rank;
};

#define SYS __HIP_MEMORY_SCOPE_SYSTEM
#define AGT __HIP_MEMORY_SCOPE_AGENT
#define WGS __HIP_MEMORY_SCOPE_WORKGROUP
#define RLX __ATOMIC_RELAXED
#define VMCNT(n) asm volatile("s_waitcnt vmcnt(" #n ")" ::: "memory")
#define VMCNTN(n) asm volatile("s_waitcnt vmcnt(%0)" :: "n"(n) : "memory")
#define VMCNT0_SEEN __builtin_amdgcn_s_waitcnt(0x0F70)
#define LGKM0 asm volatile("s_waitcnt lgkmcnt(0)" ::: "memory")
#define CFENCE asm volatile("" ::: "memory")
#define START_BARRIER do { __builtin_amdgcn_s_barrier(); CFENCE; } while (0)

#define STAMP(slot) do { } while (0)

__device__ __forceinline__ auto mk_rsrc(const void* base, unsigned bytes) {
    return __builtin_amdgcn_make_buffer_rsrc(const_cast<void*>(base), 0, bytes, 0x00020000);
}
__device__ __forceinline__ void glds4(const char* gbase, unsigned voff, unsigned lds) {
    unsigned keep;
    asm volatile("s_mov_b32 %0, m0\n\ts_mov_b32 m0, %3\n\ts_nop 0\n\t"
                 "global_load_lds_dwordx4 %1, %2 sc0 sc1\n\t"
                 "global_load_lds_dwordx4 %1, %2 offset:1024 sc0 sc1\n\t"
                 "global_load_lds_dwordx4 %1, %2 offset:2048 sc0 sc1\n\t"
                 "global_load_lds_dwordx4 %1, %2 offset:3072 sc0 sc1\n\t"
                 "s_mov_b32 m0, %0"
                 : "=&s"(keep) : "v"(voff), "s"(gbase), "s"(lds) : "memory");
}
__device__ __forceinline__ void glds1(const char* gbase, unsigned voff, unsigned lds) {
    unsigned keep;
    asm volatile("s_mov_b32 %0, m0\n\ts_mov_b32 m0, %3\n\ts_nop 0\n\t"
                 "global_load_lds_dwordx4 %1, %2\n\t"
                 "s_mov_b32 m0, %0"
                 : "=&s"(keep) : "v"(voff), "s"(gbase), "s"(lds) : "memory");
}
__device__ __forceinline__ bool expired(u64 t0) { return __builtin_amdgcn_s_memrealtime() - t0 > DEADLINE; }
__device__ __forceinline__ unsigned lds_rd(volatile LAS unsigned* a) { return __builtin_amdgcn_readfirstlane(*a); }
__device__ __forceinline__ void lds_wait_ge(volatile LAS unsigned* a, unsigned v) {
    while ((int)(lds_rd(a) - v) < 0) __builtin_amdgcn_s_sleep(1);
    CFENCE;
}
__device__ __forceinline__ void lds_spin_ge(volatile LAS unsigned* a, unsigned v) {
    while ((int)(lds_rd(a) - v) < 0) { }
    CFENCE;
}
__device__ __forceinline__ void lds_add(volatile LAS unsigned* a) {
    __hip_atomic_fetch_add((LAS unsigned*)a, 1u, RLX, WGS);
}
__device__ __forceinline__ unsigned wave_max(unsigned v) {
    v = max(v, (unsigned)__builtin_amdgcn_update_dpp(0, (int)v, 0x111, 0xf, 0xf, false));
    v = max(v, (unsigned)__builtin_amdgcn_update_dpp(0, (int)v, 0x112, 0xf, 0xf, false));
    v = max(v, (unsigned)__builtin_amdgcn_update_dpp(0, (int)v, 0x114, 0xf, 0xf, false));
    v = max(v, (unsigned)__builtin_amdgcn_update_dpp(0, (int)v, 0x118, 0xf, 0xf, false));
    return max(max((unsigned)__builtin_amdgcn_readlane((int)v, 15), (unsigned)__builtin_amdgcn_readlane((int)v, 31)),
               max((unsigned)__builtin_amdgcn_readlane((int)v, 47), (unsigned)__builtin_amdgcn_readlane((int)v, 63)));
}
__device__ __forceinline__ unsigned lane_prefix(unsigned c, unsigned* total) {
    unsigned pre = 0, tot = 0;
#pragma unroll
    for (int k = 0; k < 6; ++k) {
        const u64 m = __ballot((c >> k) & 1u);
        pre += __builtin_amdgcn_mbcnt_hi((unsigned)(m >> 32), __builtin_amdgcn_mbcnt_lo((unsigned)m, 0u)) << k;
        tot += (unsigned)__builtin_popcountll(m) << k;
    }
    *total = tot;
    return pre;
}
__device__ __forceinline__ unsigned quant4(v4f y, float inv) {
    unsigned r[4];
#pragma unroll
    for (int e = 0; e < 4; ++e) r[e] = __float_as_uint(fminf(__builtin_fmaf(y[e], inv, 12582912.0f), 12582912.0f + 127.0f));
    return __builtin_amdgcn_perm(r[1], r[0], 0x0c0c0400u) | __builtin_amdgcn_perm(r[3], r[2], 0x04000c0cu);
}
__device__ __forceinline__ unsigned wait_granule(const char* addr, unsigned* tmo, unsigned seq, u64 t0) {
    for (;;) {
        u64 f;
        asm volatile("s_load_dwordx2 %0, %1, 0x0 glc\n\ts_waitcnt lgkmcnt(0)" : "=s"(f) : "s"(addr) : "memory");
        if ((unsigned)(f >> 32) == seq) return (unsigned)f;
        const u64 now = __builtin_amdgcn_s_memrealtime();
        if (now - t0 > 50000ull) {
            if (now - t0 > DEADLINE || __hip_atomic_load(tmo, RLX, AGT) == seq) {
                __hip_atomic_store(tmo, seq, RLX, AGT);
                return 0xffffffffu;
            }
        }
        __builtin_amdgcn_s_sleep(2);
    }
}

__global__ void __launch_bounds__(NWAVE * 64, 4) fused_gemm_ar_q8(Params p) {
    extern __shared__ __attribute__((aligned(16))) char smem[];
    const int tid = threadIdx.x;
    const int lane = tid & 63;
    const int wave = __builtin_amdgcn_readfirstlane(tid >> 6);
    const int b = blockIdx.x;
    const int rank = p.rank;
    char* const myws = p.ws[rank];
    unsigned* const tmo = (unsigned*)(myws + OFF_TMO);
    volatile LAS unsigned* aux = (volatile LAS unsigned*)(smem + LDS_AUX);
    const unsigned ldsbase = (unsigned)(size_t)(LAS char*)smem;
    const int xcd = b & 7, idx = b >> 3;
    const int tm = (xcd >> 1) * 8 + (idx & 7);
    const int tn = (xcd & 1) * 4 + (idx >> 3);
    const int fm = lane & 15, fq = lane >> 4;

    if (wave < 4) {
        if (tid < 64) aux[tid] = 0u;
        const int wm = wave >> 1, wn = wave & 1;
        v4f acc[2][4];
#pragma unroll
        for (int i = 0; i < 2; ++i)
#pragma unroll
            for (int j = 0; j < 4; ++j) acc[i][j] = v4f{0.f, 0.f, 0.f, 0.f};
        const unsigned xaddr = (unsigned)wm * 2048u + 16u * (unsigned)(4 * fm + (fq ^ ((4 - (fm >> 2)) & 3)));
        unsigned waddr;
        {
            const unsigned gg = (unsigned)fq, aa = (unsigned)((lane >> 2) & 3), bb = (unsigned)(lane & 3);
            const unsigned pp = bb >> 1, beta = bb & 1;
            const unsigned pos = ((2u * beta) ^ aa) | ((gg & 1u) << 2) | (pp << 3) | (aa << 4) | ((gg >> 1) << 6);
            waddr = 4096u + ((unsigned)wn * 2u + pp) * 2048u + 16u * pos + 8u * beta;
        }
        LGKM0;
        START_BARRIER;
        if (wave == 0) STAMP(0);
        {
            static_assert(NSLOT == 12 && NUNIT == 21 * 6 + 2, "ring unrolling");
            v8s xf[2][2], wf[2][4];
            unsigned xa = ldsbase + xaddr + 6u * SLOT, wa[8];
#pragma unroll
            for (int t = 0; t < 8; ++t)
                wa[t] = ldsbase + (waddr ^ (unsigned)(((t >> 2) & 1) << 4) ^ (unsigned)(((t >> 1) & 1) << 3) ^ (unsigned)((t & 1) << 7)) + 6u * SLOT;
            unsigned fa = ldsbase + LDS_AUX + (unsigned)(A_FULL + 6) * 4u;
            const unsigned consa = ldsbase + LDS_AUX + (unsigned)(A_CONS + wave) * 4u;
            auto rd = [&](int set, int i) {
#pragma unroll
                for (int t = 0; t < 4; ++t) {
                    v4s lo = __builtin_amdgcn_ds_read_tr16_b64_v4i16((LAS v4s*)(size_t)(wa[2 * t] + (unsigned)i * SLOT));
                    v4s hi = __builtin_amdgcn_ds_read_tr16_b64_v4i16((LAS v4s*)(size_t)(wa[2 * t + 1] + (unsigned)i * SLOT));
                    wf[set][t] = __builtin_shufflevector(lo, hi, 0, 1, 2, 3, 4, 5, 6, 7);
                    if (t == 0) {
#pragma unroll
                        for (int mt = 0; mt < 2; ++mt) xf[set][mt] = *(const LAS v8s*)(size_t)(xa + (unsigned)i * SLOT + mt * 1024);
                    }
                }
            };
            auto mm = [&](int set) {
#pragma unroll
                for (int t = 0; t < 4; ++t)
#pragma unroll
                    for (int mt = 0; mt < 2; ++mt)
                        acc[mt][t] = __builtin_amdgcn_mfma_f32_16x16x32_bf16((v8bf)wf[set][t], (v8bf)xf[set][mt], acc[mt][t], 0, 0, 0);
            };
            auto step = [&](int n, int i) {
                LGKM0;
                *(volatile LAS unsigned*)(size_t)consa = (unsigned)(n + 1);
                const unsigned f = *(volatile LAS unsigned*)(size_t)(fa + (unsigned)i * 4u);
                CFENCE;
                rd((i & 1) ^ 1, i);
                mm(i & 1);
                __builtin_amdgcn_sched_group_barrier(0x100, 2, 0); __builtin_amdgcn_sched_group_barrier(0x8, 1, 0);
                __builtin_amdgcn_sched_group_barrier(0x100, 2, 0); __builtin_amdgcn_sched_group_barrier(0x8, 1, 0);
                __builtin_amdgcn_sched_group_barrier(0x100, 2, 0); __builtin_amdgcn_sched_group_barrier(0x8, 1, 0);
                __builtin_amdgcn_sched_group_barrier(0x100, 2, 0); __builtin_amdgcn_sched_group_barrier(0x8, 1, 0);
                __builtin_amdgcn_sched_group_barrier(0x100, 2, 0); __builtin_amdgcn_sched_group_barrier(0x8, 4, 0);
                __builtin_amdgcn_sched_barrier(0);
                if ((int)(__builtin_amdgcn_readfirstlane(f) - (unsigned)(n + 2)) < 0) {
                    lds_wait_ge((volatile LAS unsigned*)(size_t)(fa + (unsigned)i * 4u), (unsigned)(n + 2));
                    rd((i & 1) ^ 1, i);
                }
            };
            __builtin_amdgcn_s_setprio(2);
            lds_wait_ge(aux + A_FULL + 11, 1u);
            rd(0, 5);
            unsigned dl = 0u - 6u * SLOT, df = 0u - 24u;
            auto flip = [&]() {
                xa += dl;
#pragma unroll
                for (int t = 0; t < 8; ++t) wa[t] += dl;
                fa += df;
                dl = 0u - dl;
                df = 0u - df;
                asm volatile("" : "+v"(xa), "+v"(wa[0]), "+v"(wa[1]), "+v"(wa[2]), "+v"(wa[3]), "+v"(wa[4]), "+v"(wa[5]), "+v"(wa[6]), "+v"(wa[7]), "+v"(fa));
            };
            for (int n = 0; n < NUNIT - 2; n += 6) {
                flip();
#pragma unroll
                for (int i = 0; i < 6; ++i) step(n + i, i);
            }
            flip();
            step(NUNIT - 2, 0);
            mm(1);
        }
        if (wave == 0) STAMP(3);
        unsigned m32 = 0;
#pragma unroll
        for (int i = 31; i >= 0; --i) m32 = m32 + m32 + (acc[i >> 4][(i >> 2) & 3][i & 3] > 0.f ? 1u : 0u);
        *(LAS unsigned*)(smem + L_MASK + (unsigned)tid * 4u) = m32;
        CFENCE;
        if (lane == 0) lds_add(aux + A_CNTM);
        __builtin_amdgcn_sched_barrier(0);
        float mx = 0.f;
#pragma unroll
        for (int i = 0; i < 2; ++i)
#pragma unroll
            for (int j = 0; j < 4; ++j)
#pragma unroll
                for (int e = 0; e < 4; ++e) {
                    acc[i][j][e] = fmaxf(acc[i][j][e], 0.f);
                    mx = fmaxf(mx, acc[i][j][e]);
                }
        const unsigned wmx = wave_max(__float_as_uint(mx));
        if (lane == 0) {
            __hip_atomic_fetch_max((LAS unsigned*)(aux + A_MAX), wmx, RLX, WGS);
            CFENCE;
            lds_add(aux + A_CNT);
        }
        unsigned tot;
        const unsigned pre = lane_prefix((unsigned)__builtin_popcount(m32), &tot);
        if (lane == 0) { aux[A_TOT + wave] = tot; CFENCE; lds_add(aux + A_CNT1); }
        lds_wait_ge(aux + A_CNT1, 4u);
        const v4u tt = *(const volatile LAS v4u*)(aux + A_TOT);
        const unsigned T = __builtin_amdgcn_readfirstlane(tt.x + tt.y + tt.z + tt.w);
        {
            unsigned a = L_STREAM + 4u * (pre + (wave > 0 ? tt.x : 0u) + (wave > 1 ? tt.y : 0u) + (wave > 2 ? tt.z : 0u));
            const unsigned dummy = L_DUMMY + (unsigned)tid * 4u;
#pragma unroll
            for (int i = 0; i < 2; ++i)
#pragma unroll
                for (int j = 0; j < 4; ++j)
#pragma unroll
                    for (int e = 0; e < 4; ++e) {
                        const bool on = acc[i][j][e] > 0.f;
                        *(LAS float*)(smem + (on ? a : dummy)) = acc[i][j][e];
                        a += on ? 4u : 0u;
                    }
        }
        CFENCE;
        if (lane == 0) lds_add(aux + A_CNT2);
        lds_wait_ge(aux + A_CNT2, 4u);
        v4f sv[2][4];
#pragma unroll
        for (int h = 0; h < 2; ++h)
#pragma unroll
            for (int j = 0; j < 4; ++j)
                sv[h][j] = *(const LAS v4f*)(smem + L_STREAM + (unsigned)((wave + 4 * h) * 4096 + lane * 64 + j * 16));
        lds_spin_ge(aux + A_GO, 1u);
        if (wave == 0) STAMP(4);
        const float amax = __uint_as_float(aux[A_GM]);
        const bool dead = aux[A_DEAD] != 0u;
        const float inv = amax > 0.f ? 127.0f / amax : 0.f;
        const v4u bad = {0x80808080u, 0x80808080u, 0x80808080u, 0x80808080u};
#pragma unroll
        for (int h = 0; h < 2; ++h) {
            if (h == 0 || T > 4096u) {
                const unsigned c = (unsigned)((wave + 4 * h) * 64 + lane);
                const unsigned q0 = quant4(sv[h][0], inv), q1 = quant4(sv[h][1], inv), q2 = quant4(sv[h][2], inv), q3 = quant4(sv[h][3], inv);
                LAS unsigned* const dst = (LAS unsigned*)(smem + L_PACK + c * 12u);
                if (c * 16u < T) {
                    dst[0] = (q0 & 0x3f3f3f3fu) | ((q3 << 6) & 0xc0c0c0c0u);
                    dst[1] = (q1 & 0x3f3f3f3fu) | ((q3 << 4) & 0xc0c0c0c0u);
                    dst[2] = (q2 & 0x3f3f3f3fu) | ((q3 << 2) & 0xc0c0c0c0u);
                    unsigned m = 0;
                    if ((q0 | q1 | q2 | q3) & 0x40404040u) {
                        m = ((((q0 >> 6) & 0x01010101u) * 0x10204080u) >> 28) | (((((q1 >> 6) & 0x01010101u) * 0x10204080u) >> 28) << 4)
                          | (((((q2 >> 6) & 0x01010101u) * 0x10204080u) >> 28) << 8) | (((((q3 >> 6) & 0x01010101u) * 0x10204080u) >> 28) << 12);
                        unsigned e = __hip_atomic_fetch_add((LAS unsigned*)(aux + A_EXC), (unsigned)__builtin_popcount(m), RLX, WGS);
                        for (unsigned r = m; r && e < 512u; r &= r - 1u, ++e)
                            *(LAS unsigned short*)(smem + L_LIST + 2u * e) = (unsigned short)(c * 16u + (unsigned)__builtin_ctz(r));
                    }
                    *(LAS unsigned short*)(smem + L_PLANE + 2u * c) = (unsigned short)m;
                }
            }
        }
        CFENCE;
        if (lane == 0) lds_add(aux + A_CNT3);
        auto ro = mk_rsrc(p.out[rank], (unsigned)NCOL * 4096u);
        const unsigned ooff = (unsigned)((rank >> 2) * 2048 + tm * 64 + wm * 32 + fm) * NCOL + (unsigned)((rank & 3) * 1024 + tn * 128 + wn * 64 + fq * 16);
#pragma unroll
        for (int mt = 0; mt < 2; ++mt) {
            const unsigned d0 = quant4(acc[mt][0], inv), d1 = quant4(acc[mt][1], inv), d2 = quant4(acc[mt][2], inv), d3 = quant4(acc[mt][3], inv);
            const bool odd = fq & 1;
            v4u qv = {odd ? d1 : d0, odd ? d0 : d1, odd ? d3 : d2, odd ? d2 : d3};
            if (dead) qv = bad;
            __builtin_amdgcn_raw_buffer_store_b128(qv, ro, ooff + mt * 16 * NCOL, 0, 16);
        }
        return;
    }

    unsigned* const ctr = (unsigned*)(myws + OFF_CTR) + (b * 16 + wave);
    const unsigned seqv = __hip_atomic_load(ctr, RLX, AGT) + 1u;
    const int q = b & 31;
    const int dest = (rank + (b >> 5)) & 7;

    unsigned* const prog = (unsigned*)(p.ws[dest] + OFF_PROG + (unsigned)((lane & (FREP - 1)) * 32 + q) * 128u + (unsigned)rank * 4u);

    const unsigned xo = (unsigned)((dest >> 2) * 2048 + q * 64 + (lane >> 2)) * (KC * 2) + (unsigned)((lane & 3) ^ ((4 - (lane >> 4)) & 3)) * 16u;
    const unsigned wo = (unsigned)((lane >> 4) | ((((lane >> 3) & 1) ^ (q & 1)) << 2) | (((lane >> 2) & 1) << 3)) * (NCOL * 2)
                      + (unsigned)((dest & 3) * 1024 + q * 32) * 2u + (unsigned)((lane & 3) ^ (lane >> 4)) * 16u;
    int st;
    unsigned seq;

    if (wave == W_COPY || wave == W_COPY + 1) {
        st = wave - W_COPY;
        __builtin_amdgcn_s_setprio(3);
        v4u d[3];
        d[0] = *(const v4u*)(p.x + xo + (unsigned)(st * 16 * (KC * 2)));
        d[1] = *(const v4u*)(p.x + xo + (unsigned)((st + 2) * 16 * (KC * 2)));
        d[2] = *(const v4u*)(p.w + wo + (unsigned)(st * 16 * (NCOL * 2)));
        if (st == 0) STAMP(8);
        auto rs = mk_rsrc(p.ws[dest] + OFF_STAGE + (size_t)rank * NSUB * UNIT, NSUB * UNIT);
        const unsigned ox = (unsigned)q * 4096u + (unsigned)(st * 1024 + lane * 16);
        const unsigned ow = UNIT_X + (unsigned)q * 2048u + (unsigned)(st * 1024 + lane * 16);
        const LAS char* const src = (const LAS char*)smem + LDS_CPY + (unsigned)(st * 1024 + lane * 16);
        const unsigned pace = dest != rank ? PACE : 0u;
        VMCNT0_SEEN;
        const unsigned tc = (unsigned)__builtin_amdgcn_s_memrealtime() - (xcd == 0 ? XADV : 0u);
        int grp = 0;
        for (int s = 0; s < NSUB; ++s) {
            if (s > 0) {
                lds_wait_ge(aux + A_FETCHED, (unsigned)s);
#pragma unroll
                for (int j = 0; j < 3; ++j) d[j] = *(const LAS v4u*)(src + grp * 6144 + j * 2048);
                grp = grp == CG - 1 ? 0 : grp + 1;
                LGKM0;
                if (lane == 0) aux[A_TAKEN + st] = (unsigned)s;
            }
#pragma unroll
            for (int j = 0; j < 3; ++j) {
                VMCNT(1);
                if (j == 1 && s > 0 && lane == 0) aux[A_SENT + st] = (unsigned)s;
                const unsigned due = (unsigned)(s * 6 + j * 2 + st) * pace / 6u;
                while ((unsigned)__builtin_amdgcn_s_memrealtime() - tc < due) __builtin_amdgcn_s_sleep(1);
                __builtin_amdgcn_raw_buffer_store_b128(d[j], rs, (j < 2 ? ox + j * 2048 : ow) + (unsigned)s * UNIT, 0, 17);
                if (s == 0 && j == 0) { if (st == 0) STAMP(9); START_BARRIER; }
            }
        }
        VMCNT(0);
        seq = __builtin_amdgcn_readfirstlane(seqv);
        unsigned old = 0;
        if (lane == 0) old = __hip_atomic_fetch_add((LAS unsigned*)(aux + A_CDONE), 1u, RLX, WGS);
        if (__builtin_amdgcn_readfirstlane(old) == 1u && lane < FREP) __hip_atomic_fetch_max(prog, seq * 32u + (unsigned)NSUB, RLX, SYS);
        if (lane == 0) __hip_atomic_store(ctr, seq, RLX, AGT);
        STAMP(10);
    } else {

    if (wave == W_FETCH) {
        auto fetch = [&](int g) {
            const unsigned l0 = ldsbase + LDS_CPY + (unsigned)(g % CG) * 6144u;
#pragma unroll
            for (int j = 0; j < 4; ++j) glds1(p.x, xo + (unsigned)((g + 1) * 64 + j * 16 * (KC * 2)), l0 + j * 1024u);
#pragma unroll
            for (int j = 0; j < 2; ++j) glds1(p.w, wo + (unsigned)(((g + 1) * 32 + j * 16) * (NCOL * 2)), l0 + 4096u + j * 1024u);
        };
        constexpr int NG = NSUB - 1;
#pragma unroll
        for (int g = 0; g < CG; ++g) fetch(g);
        START_BARRIER;
#pragma unroll
        for (int g = 0; g < NG; ++g) {
            VMCNTN(6 * ((CG + g < NG ? CG + g : NG) - g - 1));
            if (lane == 0) aux[A_FETCHED] = (unsigned)(g + 1);
            if (g + CG < NG) { lds_wait_ge(aux + A_TAKEN, (unsigned)(g + 1)); lds_wait_ge(aux + A_TAKEN + 1, (unsigned)(g + 1)); fetch(g + CG); }
        }
        VMCNT0_SEEN;
        seq = __builtin_amdgcn_readfirstlane(seqv);
        if (lane == 0) __hip_atomic_store(ctr, seq, RLX, AGT);
        const int dst = (rank + 1 + ((lane < 7 ? lane : 0) + b % 7) % 7) & 7;
        u64* const fl = (u64*)(p.ws[dst] + OFF_P2F + (unsigned)b * 128u + (unsigned)rank * 8u);
        lds_wait_ge(aux + A_CNT3, 4u);
        const u64 fv = ((u64)seq << 32) | (aux[A_DEAD] ? 0xffffffffu : aux[A_EXC]);
        int done = 0;
        while (done < 6) {
            int d2;
            for (;;) {
                const v4u pp = *(const volatile LAS v4u*)(aux + A_P2);
                d2 = (int)__builtin_amdgcn_readfirstlane(min(pp.x, pp.y));
                if (d2 > done) break;
                __builtin_amdgcn_s_sleep(1);
            }
            if (lane >= done && lane < d2 && lane < 6) __hip_atomic_store(fl, fv, RLX, SYS);
            done = d2;
        }
        return;
    }

    seq = __builtin_amdgcn_readfirstlane(seqv);
    if (lane == 0) __hip_atomic_store(ctr, seq, RLX, AGT);
    const u64 t0 = __builtin_amdgcn_s_memrealtime();
    VMCNT0_SEEN;
    START_BARRIER;

    if (wave == W_POLL) {
        const char* px = myws + OFF_PROG + (unsigned)tm * 128u;
        const char* pw = myws + OFF_PROG + (unsigned)tn * 512u;
        const unsigned r8 = (unsigned)(((idx & 7) ^ (((idx >> 3) << 1) | (xcd & 1))) & (FREP - 1));
        unsigned radd = (r8 & (PREP - 1)) * 4096u, radd2 = (r8 & ~(PREP - 1)) * 4096u;
        int pvk = 0;
        const unsigned pbase = seq * 32u;
        unsigned sent = 0;
        bool stop = false, fin = false, fast = false;
        while (!fin) {
            const u64 now = __builtin_amdgcn_s_memrealtime();
            u64 gap = 0;
            if (!stop) {
                int pv = 0, lo = NSUB;
                v8u fx, f0, f1, f2, f3;
                asm volatile("s_load_dwordx8 %0, %5, 0x0 glc\n\ts_load_dwordx8 %1, %6, 0x0 glc\n\ts_load_dwordx8 %2, %6, 0x80 glc\n\t"
                             "s_load_dwordx8 %3, %6, 0x100 glc\n\ts_load_dwordx8 %4, %6, 0x180 glc\n\ts_waitcnt lgkmcnt(0)"
                             : "=&s"(fx), "=&s"(f0), "=&s"(f1), "=&s"(f2), "=&s"(f3) : "s"(px), "s"(pw) : "memory");
#pragma unroll
                for (int c = 0; c < 8; ++c) {
                    const int m = min(min(min((int)(f0[c] - pbase), (int)(f1[c] - pbase)), min((int)(f2[c] - pbase), (int)(f3[c] - pbase))), (int)(fx[c] - pbase));
                    lo = min(lo, m);
                    pv = lane == c ? m : pv;
                }
                pv = pvk = max(pv, pvk);
                if (lo >= (int)FINE_FROM) { px += radd; pw += radd; radd = 0; fast = true; }
                if (lo >= NSUB - 1) { px += radd2; pw += radd2; radd2 = 0; }
                stop = lo >= NSUB;
                if (!stop && now - t0 > 50000ull) {
                    if (now - t0 > DEADLINE || __hip_atomic_load(tmo, RLX, AGT) == seq) {
                        __hip_atomic_store(tmo, seq, RLX, AGT);
                        pv = NSUB;
                        stop = true;
                        if (lane == 0) aux[A_DEAD] = 1u;
                    }
                }
                if (lane < 8) aux[A_PRG + lane] = (unsigned)pv;
                if (stop) STAMP(15);
                gap = fast ? GAP_FAST : GAP_SLOW;
            }
            do {
                if (lds_rd(aux + A_CDONE) == 2u) { if (stop) { fin = true; break; } }
                else {
                    const unsigned d = min(lds_rd(aux + A_SENT), lds_rd(aux + A_SENT + 1));
                    if (d >= sent + (d > FINE_FROM ? 1u : FSTEP)) {
                        sent = d;
                        if (lane < (d > FINE_FROM ? PREP : 1)) __hip_atomic_fetch_max(prog, pbase + d, RLX, SYS);
                    }
                }
                __builtin_amdgcn_s_sleep(1);
            } while (__builtin_amdgcn_s_memrealtime() - now < gap);
        }
        return;
    } else if (wave == W_SCHED) {
        const int csrc = (rank + lane) & 7;
        int nxt = lane < 8 ? 0 : NSUB;
        int asg = 0;
        while (asg < NUNIT) {
            const int prg = (int)aux[A_PRG + csrc];
            unsigned av = (unsigned)__ballot(prg > nxt);
            if (av == 0) { __builtin_amdgcn_s_sleep(1); continue; }
            while (av) {
                const int k = __builtin_popcount(av & ((1u << (lane & 7)) - 1u));
                if (prg > nxt) { aux[A_DESC + asg + k] = (unsigned)(csrc * NSUB + nxt) * UNIT; ++nxt; }
                asg += __builtin_popcount(av);
                CFENCE;
                if (lane == 0) aux[A_ASG] = (unsigned)asg;
                av = (unsigned)__ballot(prg > nxt);
            }
        }
        STAMP(23);
        lds_wait_ge(aux + A_CNT, 4u);
        if (lane == 0) __hip_atomic_store((u64*)(myws + OFF_WGMAX + (unsigned)b * 8u), ((u64)seq << 32) | aux[A_MAX], RLX, AGT);
        bool dead = __hip_atomic_load(tmo, RLX, AGT) == seq;
        if (b == 0) {
            auto rg = mk_rsrc(myws + OFF_WGMAX, NWG * 8);
            unsigned mx;
            for (;;) {
                v4u g0 = __builtin_amdgcn_raw_buffer_load_b128(rg, (unsigned)lane * 32u, 0, 17);
                v4u g1 = __builtin_amdgcn_raw_buffer_load_b128(rg, (unsigned)lane * 32u + 16u, 0, 17);
                unsigned tm_ = __hip_atomic_load(tmo, RLX, AGT);
                bool ok = g0.y == seq && g0.w == seq && g1.y == seq && g1.w == seq;
                mx = max(max(g0.x, g0.z), max(g1.x, g1.z));
                if (__all(ok)) break;
                if (tm_ == seq || expired(t0)) { __hip_atomic_store(tmo, seq, RLX, AGT); dead = true; break; }
            }
            mx = wave_max(mx);
            char* dst = p.ws[(lane >> 2) & 7] + OFF_RMAX + (unsigned)(lane & 3) * 512u + (unsigned)rank * 8u;
            if (lane < 32) __hip_atomic_store((u64*)dst, ((u64)seq << 32) | mx, RLX, SYS);
            if (lane == 0) aux[A_LEAD] = 1u;
            STAMP(7);
            const char* const ra = myws + OFF_RMAX;
            unsigned seen = 0, flip = 0;
            mx = 0;
            for (;;) {
                v16u g0, g1;
                asm volatile("s_load_dwordx16 %0, %2, 0x0 glc\n\ts_load_dwordx16 %1, %2, 0x200 glc\n\ts_waitcnt lgkmcnt(0)"
                             : "=&s"(g0), "=&s"(g1) : "s"(ra + flip) : "memory");
                flip ^= 1024u;
#pragma unroll
                for (int c = 0; c < 8; ++c) {
                    if (g0[2 * c + 1] == seq) { seen |= 1u << c; mx = max(mx, g0[2 * c]); }
                    if (g1[2 * c + 1] == seq) { seen |= 1u << c; mx = max(mx, g1[2 * c]); }
                }
                if (seen == 255u) break;
                const u64 now = __builtin_amdgcn_s_memrealtime();
                if (now - t0 > 50000ull) {
                    if (now - t0 > DEADLINE || __hip_atomic_load(tmo, RLX, AGT) == seq) { __hip_atomic_store(tmo, seq, RLX, AGT); dead = true; break; }
                }
            }
            if (lane < 32) __builtin_amdgcn_raw_buffer_store_b128(v4u{mx, seq, (unsigned)__builtin_amdgcn_s_memrealtime(), seq}, mk_rsrc(myws + OFF_GMAX, 4096u), (unsigned)lane * 128u, 0, 16);
            STAMP(14);
        }
        const char* const ga = myws + OFF_GMAX + (unsigned)(b & 31) * 128u;
        unsigned gm, tg;
        for (;;) {
            v4u g;
            asm volatile("s_load_dwordx4 %0, %1, 0x0 glc\n\ts_waitcnt lgkmcnt(0)" : "=s"(g) : "s"(ga) : "memory");
            gm = g.x;
            tg = g.z;
            if (g.y == seq && g.w == seq) break;
            const u64 now = __builtin_amdgcn_s_memrealtime();
            if (now - t0 > 50000ull) {
                if (now - t0 > DEADLINE || __hip_atomic_load(tmo, RLX, AGT) == seq) { __hip_atomic_store(tmo, seq, RLX, AGT); dead = true; break; }
            }
        }
        if (dead) tg = (unsigned)__builtin_amdgcn_s_memrealtime();
        if (lane == 0) {
            aux[A_GM] = gm;
            aux[A_TGO] = tg;
            if (dead) aux[A_DEAD] = 1u;
            CFENCE;
            aux[A_GO] = 1u;
        }
        return;
    } else {
        const int l = wave - 4;
        const char* const stage = myws + OFF_STAGE;
        const unsigned vx = (unsigned)tm * 4096u + (unsigned)lane * 16u;
        const unsigned vw = UNIT_X + (unsigned)tn * 8192u + (unsigned)lane * 16u;
        auto issue = [&](int n, int sl) {
            const char* gb = stage + lds_rd(aux + A_DESC + n);
            const unsigned ld = ldsbase + (unsigned)sl * SLOT;
            glds4(gb, vx, ld);
            glds4(gb, vw, ld + 4096u);
            glds4(gb + 4096, vw, ld + 8192u);
        };
        auto ready = [&](int n) {
            const unsigned as = aux[A_ASG];
            const v4u cs = *(const volatile LAS v4u*)(aux + A_CONS);
            const bool r = __builtin_amdgcn_readfirstlane(min(as, min(min(cs.x, cs.y), min(cs.z, cs.w)) + NSLOT)) > (unsigned)n;
            CFENCE;
            return r;
        };
        int slot = l == 0 ? NSLOT - 1 : l - 1;
        while (!ready(l)) __builtin_amdgcn_s_sleep(1);
        issue(l, slot);
        for (int n = l;;) {
            const int nn = n + NL, ns = slot + NL >= NSLOT ? slot + NL - NSLOT : slot + NL;
            const bool next = nn < NUNIT && ready(nn);
            if (next) { issue(nn, ns); VMCNT(12); } else VMCNT(0);
            if (lane == 0) aux[A_FULL + slot] = (unsigned)(n + 1);
            if (nn >= NUNIT) break;
            if (!next) { while (!ready(nn)) __builtin_amdgcn_s_sleep(1); issue(nn, ns); }
            n = nn;
            slot = ns;
        }
        const int src = (rank + 1 + l) & 7;
        const char* const fl = myws + OFF_P2F + (unsigned)b * 128u + (unsigned)src * 8u;
        auto rp = mk_rsrc(myws + OFF_P2 + (unsigned)(src * NWG + b) * P2_PITCH, P2_PITCH);
        lds_wait_ge(aux + A_GO, 1u);
        {
            const unsigned tg = lds_rd(aux + A_TGO);
            while ((unsigned)__builtin_amdgcn_s_memrealtime() - tg < MWAIT) __builtin_amdgcn_s_sleep(2);
        }
        unsigned mk[4];
#pragma unroll
        for (int w = 0; w < 4; ++w) mk[w] = __builtin_amdgcn_raw_buffer_load_b32(rp, (unsigned)(w * 256 + lane * 4), 0, 17);
        unsigned nexc = 0;
        unsigned sel[8][6], off[8];
        v3u pk[8];
        unsigned ex0, xi, nch;
        int nit;
        for (int pass = 0;; ++pass) {
            unsigned base = 0;
#pragma unroll
            for (int w = 0; w < 4; ++w) {
                unsigned tot;
                const unsigned pre = lane_prefix((unsigned)__builtin_popcount(mk[w]), &tot);
                off[2 * w] = base + pre;
                off[2 * w + 1] = base + pre + (unsigned)__builtin_popcount(mk[w] & 0xffffu);
                base += tot;
#pragma unroll
                for (int mt = 0; mt < 2; ++mt) {
                    const unsigned m16 = (mk[w] >> (16 * mt)) & 0xffffu;
#pragma unroll
                    for (int t = 0; t < 4; ++t) {
                        const unsigned bits = (((m16 >> (4 * t)) & 15u) * 0x00204081u) & 0x01010101u;
                        const unsigned ix = bits * 0x01010100u + (unsigned)__builtin_popcount(m16 & ((1u << (4 * t)) - 1u)) * 0x01010101u;
                        if (t < 2) {
                            const unsigned ma = bits * 0xffu;
                            sel[2 * w + mt][t] = (ix & ma) | (0x0c0c0c0cu & ~ma);
                        } else {
                            const unsigned hi = (ix >> 3) & bits;
                            const unsigned ma = (bits ^ hi) * 0xffu, mb = hi * 0xffu;
                            sel[2 * w + mt][t] = (ix & ma) | (0x0c0c0c0cu & ~ma);
                            sel[2 * w + mt][t + 2] = (ix & 0x07070707u & mb) | (0x0c0c0c0cu & ~mb);
                        }
                    }
                }
            }
#pragma unroll
            for (int i = 0; i < 8; ++i)
                asm volatile("" : "+v"(sel[i][0]), "+v"(sel[i][1]), "+v"(sel[i][2]), "+v"(sel[i][3]), "+v"(sel[i][4]), "+v"(sel[i][5]), "+v"(off[i]));
            if (pass == 0) {
                if (l == 0) STAMP(12);
                nexc = wait_granule(fl, tmo, seq, t0);
                if (l == 0) STAMP(13);
            }
            nit = (int)((base + 1023u) >> 10);
#pragma unroll
            for (int i = 0; i < 8; ++i)
                if (i < nit) pk[i] = __builtin_amdgcn_raw_buffer_load_b96(rp, 1024u + (unsigned)(i * 64 + lane) * 12u, 0, 17);
            nch = (base + 15u) >> 4;
            ex0 = 1024u + ((nch * 12u + 15u) & ~15u);
            xi = __builtin_amdgcn_raw_buffer_load_b16(rp, ex0 + 2u * (unsigned)lane, 0, 17);
            if (pass > 0) break;
            bool same = true;
#pragma unroll
            for (int w = 0; w < 4; ++w) {
                const unsigned m2 = __builtin_amdgcn_raw_buffer_load_b32(rp, (unsigned)(w * 256 + lane * 4), 0, 17);
                same = same && m2 == mk[w];
                mk[w] = m2;
            }
            if (__all(same)) break;
        }
        const bool ok = nexc != 0xffffffffu;
        LAS char* const dec = (LAS char*)smem + L_DEC + (unsigned)l * DEC_PITCH;
        {
            const bool plane = ok && nexc > nch;
#pragma unroll
            for (int i = 0; i < 8; ++i) {
                if (i < nit) {
                    v4u q = {pk[i].x & 0x3f3f3f3fu, pk[i].y & 0x3f3f3f3fu, pk[i].z & 0x3f3f3f3fu,
                             ((pk[i].x >> 6) & 0x03030303u) | ((pk[i].y >> 4) & 0x0c0c0c0cu) | ((pk[i].z >> 2) & 0x30303030u)};
                    if (plane) {
                        const unsigned m = __builtin_amdgcn_raw_buffer_load_b16(rp, ex0 + (unsigned)(i * 64 + lane) * 2u, 0, 17);
#pragma unroll
                        for (int t = 0; t < 4; ++t) q[t] |= ((((m >> (4 * t)) & 15u) * 0x00204081u) & 0x01010101u) << 6;
                    }
                    *(LAS v4u*)(dec + (unsigned)(i * 1024 + lane * 16)) = q;
                }
            }
            CFENCE;
            for (unsigned e = (unsigned)lane; ok && !plane && e < nexc; e += 64u) {
                if (e >= 64u) xi = __builtin_amdgcn_raw_buffer_load_b16(rp, ex0 + 2u * e, 0, 17);
                __hip_atomic_fetch_or((LAS unsigned*)(dec + (xi & 0x1ffcu)), 0x40u << (8u * (xi & 3u)), RLX, WGS);
            }
            CFENCE;
        }
        auto ro = mk_rsrc(p.out[rank], (unsigned)NCOL * 4096u);
        const unsigned ooff = (unsigned)((src >> 2) * 2048 + tm * 64 + fm) * NCOL + (unsigned)((src & 3) * 1024 + tn * 128 + fq * 16);
        const bool odd = fq & 1;
#pragma unroll
        for (int s = 0; s < 8; ++s) {
            const LAS unsigned* const wp = (const LAS unsigned*)(dec + (off[s] & ~3u));
            const unsigned r0 = wp[0], r1 = wp[1], r2 = wp[2], r3 = wp[3], r4 = wp[4];
            const unsigned w0 = __builtin_amdgcn_alignbyte(r1, r0, off[s]), w1 = __builtin_amdgcn_alignbyte(r2, r1, off[s]);
            const unsigned w2 = __builtin_amdgcn_alignbyte(r3, r2, off[s]), w3 = __builtin_amdgcn_alignbyte(r4, r3, off[s]);
            const unsigned d0 = __builtin_amdgcn_perm(w1, w0, sel[s][0]);
            const unsigned d1 = __builtin_amdgcn_perm(w1, w0, sel[s][1]);
            const unsigned d2 = __builtin_amdgcn_perm(w1, w0, sel[s][2]) | __builtin_amdgcn_perm(w3, w2, sel[s][4]);
            const unsigned d3 = __builtin_amdgcn_perm(w1, w0, sel[s][3]) | __builtin_amdgcn_perm(w3, w2, sel[s][5]);
            v4u qv = {odd ? d1 : d0, odd ? d0 : d1, odd ? d3 : d2, odd ? d2 : d3};
            if (!ok) qv = v4u{0x80808080u, 0x80808080u, 0x80808080u, 0x80808080u};
            __builtin_amdgcn_raw_buffer_store_b128(qv, ro, ooff + (unsigned)(((s >> 2) * 32 + (s & 1) * 16) * NCOL + ((s >> 1) & 1) * 64), 0, 16);
        }
        STAMP(16 + l);
        return;
    }
    }

    const unsigned poff = OFF_P2 + (unsigned)(rank * NWG + b) * P2_PITCH;
    const int rot = b % 7;
    lds_wait_ge(aux + A_ASG, (unsigned)NUNIT);
    lds_spin_ge(aux + A_CNTM, 4u);
    const unsigned tm0 = (unsigned)__builtin_amdgcn_s_memrealtime();
    {
        const v4u mk = *(const LAS v4u*)(smem + L_MASK + (unsigned)lane * 16u);
        if (b == 0) lds_wait_ge(aux + A_LEAD, 1u);
        for (int k = st; k < 7; k += 2) {
            const int dst = (rank + 1 + (k + rot) % 7) & 7;
            VMCNT(1);
            if (b != 0) while ((unsigned)__builtin_amdgcn_s_memrealtime() - tm0 < (unsigned)k * MPACE) __builtin_amdgcn_s_sleep(1);
            __builtin_amdgcn_raw_buffer_store_b128(mk, mk_rsrc(p.ws[dst] + poff, 1024u), (unsigned)lane * 16u, 0, 17);
        }
    }
    if (st == 0) STAMP(11);
    int last = 0, pub = 0;
    lds_spin_ge(aux + A_CNT3, 4u);
    CFENCE;
    const v4u tt = *(const volatile LAS v4u*)(aux + A_TOT);
    const unsigned nch = __builtin_amdgcn_readfirstlane((tt.x + tt.y + tt.z + tt.w + 15u) >> 4), nex = lds_rd(aux + A_EXC);
    const unsigned Vp = (nch * 12u + 15u) & ~15u;
    const unsigned xadd = (nex <= nch ? L_LIST : L_PLANE) - L_PACK - Vp;
    const unsigned Tr = (Vp + 2u * min(nex, nch) + 63u) & ~63u;
    const int np = (int)((Tr + 1023u) >> 10);
    const unsigned ps = np ? (((Tr >> 6) + (unsigned)np - 1u) / (unsigned)np) << 6 : 0u;
    unsigned tv = lds_rd(aux + A_TGO);
    if ((int)(tm0 + 7u * MPACE - tv) > 0) tv = tm0 + 7u * MPACE;
    for (int k = 0, j = st; ; j += 2) {
        while (j >= np && k < 7) { j -= np; ++k; }
        if (k >= 7) break;
        const unsigned o = (unsigned)j * ps;
        const unsigned ol = o + (unsigned)lane * 16u;
        const v4u pv = *(const LAS v4u*)(smem + L_PACK + ol + (ol >= Vp ? xadd : 0u));
        char* const dp = p.ws[(rank + 1 + (k + rot) % 7) & 7] + poff + 1024u + o;
        VMCNT(1);
        if (last > pub) {
            if (lane == 0) aux[A_P2 + st] = (unsigned)last;
            pub = last;
        }
        const unsigned due = (((unsigned)k * Tr + o) * VPACE) >> 10;
        while ((int)((unsigned)__builtin_amdgcn_s_memrealtime() - tv - due) < 0) __builtin_amdgcn_s_sleep(1);
        __builtin_amdgcn_raw_buffer_store_b128(pv, mk_rsrc(dp, o < Tr ? min(ps, Tr - o) : 0u), (unsigned)lane * 16u, 0, 17);
        last = k;
    }
    VMCNT(0);
    {
        unsigned old = 0;
        if (lane == 0) old = __hip_atomic_fetch_add((LAS unsigned*)(aux + A_PDONE), 1u, RLX, WGS);
        if (__builtin_amdgcn_readfirstlane(old) == 1u && lane == 0)
            __hip_atomic_store((u64*)(p.ws[(rank + 1 + (6 + rot) % 7) & 7] + OFF_P2F + (unsigned)b * 128u + (unsigned)rank * 8u),
                               ((u64)seq << 32) | (aux[A_DEAD] ? 0xffffffffu : nex), RLX, SYS);
    }
    if (lane == 0) aux[A_P2 + st] = 7u;
    STAMP(5);
}

}

#define HIP_CHECK(expr) do { hipError_t _e = (expr); if (_e != hipSuccess) { \
    fprintf(stderr, "HIP error %s at %s:%d\n", hipGetErrorString(_e), __FILE__, __LINE__); \
    return; } } while (0)

extern "C" void kernel_launch(int rank, int nranks, ncclComm_t comm, hipStream_t stream,
                              void* const* d_in, const int* in_sizes, int n_in,
                              void* d_out, int out_size, const int* in_dtypes, int out_dtype,
                              void* const* d_out_all, void* const* d_ws_all, size_t ws_bytes) {
    (void)comm; (void)in_sizes; (void)n_in; (void)d_out; (void)in_dtypes; (void)out_dtype;
    if (nranks != NR || (unsigned)rank >= (unsigned)NR || ws_bytes < WS_NEED || out_size != NCOL * 4096) {
        fprintf(stderr, "unexpected problem geometry\n");
        return;
    }
    static bool attr_done[NR] = {false};
    if (!attr_done[rank]) {
        HIP_CHECK(hipFuncSetAttribute((const void*)fused_gemm_ar_q8, hipFuncAttributeMaxDynamicSharedMemorySize, LDS_BYTES));
        attr_done[rank] = true;
    }
    Params p;
    for (int r = 0; r < NR; ++r) { p.ws[r] = (char*)d_ws_all[r]; p.out[r] = (char*)d_out_all[r]; }
    p.x = (const char*)d_in[0];
    p.w = (const char*)d_in[1];
    p.rank = rank;
    hipLaunchKernelGGL(fused_gemm_ar_q8, dim3(NWG), dim3(NWAVE * 64), LDS_BYTES, stream, p);
    HIP_CHECK(hipGetLastError());
}
```

```cpp
#include <hip/hip_runtime.h>
#include <rccl/rccl.h>
#include <cstdio>

#define PACE 342u
#define MPACE 66u
#define VPACE 66u
#define MWAIT 320u
#define GAP_SLOW 100u
#define GAP_FAST 30u
#define TGO_ADD 80u
#define XADV 80u

namespace {

typedef short v4s __attribute__((ext_vector_type(4)));
typedef short v8s __attribute__((ext_vector_type(8)));
typedef __bf16 v8bf __attribute__((ext_vector_type(8)));
typedef float v4f __attribute__((ext_vector_type(4)));
typedef unsigned v4u __attribute__((ext_vector_type(4)));
typedef unsigned v3u __attribute__((ext_vector_type(3)));
typedef unsigned v8u __attribute__((ext_vector_type(8)));
typedef unsigned v16u __attribute__((ext_vector_type(16)));
typedef unsigned long long u64;
#define LAS __attribute__((address_space(3)))

constexpr int NR = 8;
constexpr int NCOL = 4096;
constexpr int KC = 512;
constexpr int NSUB = 16;
constexpr int NUNIT = NR * NSUB;
constexpr int NWG = 256;
constexpr int NL = 7;
constexpr int W_SCHED = 4 + NL, W_COPY = 5 + NL  , W_FETCH = 7 + NL, W_POLL = 8 + NL, NWAVE = 9 + NL;
constexpr int FREP = 4, PREP = 2;
constexpr unsigned FINE_FROM = 12, FSTEP = 2;
constexpr unsigned UNIT_X = 2048u * 32u * 2u;
constexpr unsigned UNIT_W = 32u * 1024u * 2u;
constexpr unsigned UNIT = UNIT_X + UNIT_W;
constexpr unsigned OFF_CTR = 0;
constexpr unsigned OFF_TMO = 32768;
constexpr unsigned OFF_WGMAX = 40960;
constexpr unsigned OFF_RMAX = 45056;
constexpr unsigned OFF_GMAX = 49152;
constexpr unsigned OFF_DIAG = 65536;
constexpr unsigned OFF_PROG = 262144;
constexpr unsigned OFF_P2F = 327680;
constexpr unsigned OFF_STAGE = 1u << 20;
constexpr unsigned P2_PITCH = 10240;
constexpr unsigned OFF_P2 = OFF_STAGE + NR * NSUB * UNIT;
constexpr unsigned WS_NEED = OFF_P2 + NR * NWG * P2_PITCH;
constexpr unsigned SLOT = 12288;
constexpr int NSLOT = 12;
constexpr unsigned LDS_AUX = NSLOT * SLOT;
constexpr unsigned LDS_CPY = LDS_AUX + 768;
constexpr int CG = 2;
constexpr unsigned L_MASK = LDS_CPY + CG * 6144;
constexpr unsigned LDS_BYTES = L_MASK + 1024;
constexpr unsigned L_STREAM = 0, L_PACK = 36864, L_LIST = 45056, L_PLANE = 47104, L_DUMMY = 49152, L_DEC = 61440, DEC_PITCH = 8448;
constexpr int A_ASG = 0, A_CDONE = 1, A_SENT = 2, A_CONS = 4, A_MAX = 8, A_CNT = 9, A_CNT2 = 10, A_CNT3 = 11, A_GM = 12, A_DEAD = 13,
              A_GO = 14, A_CNT1 = 15, A_FULL = 32  , A_TOT = 44, A_FETCHED = 48, A_EXC = 49, A_TAKEN = 50, A_P2 = 52  , A_TGO = 16,
              A_LEAD = 17, A_CNTM = 18, A_PDONE = 19, A_PRG = 56, A_DESC = 64;
constexpr u64 DEADLINE = 100000000ull;

struct Params {
    char* ws[NR];
    char* out[NR];
    const char* x;
    const char* w;
    int rank;
};

#define SYS __HIP_MEMORY_SCOPE_SYSTEM
#define AGT __HIP_MEMORY_SCOPE_AGENT
#define WGS __HIP_MEMORY_SCOPE_WORKGROUP
#define RLX __ATOMIC_RELAXED
#define VMCNT(n) asm volatile("s_waitcnt vmcnt(" #n ")" ::: "memory")
#define VMCNTN(n) asm volatile("s_waitcnt vmcnt(%0)" :: "n"(n) : "memory")
#define VMCNT0_SEEN __builtin_amdgcn_s_waitcnt(0x0F70)
#define LGKM0 asm volatile("s_waitcnt lgkmcnt(0)" ::: "memory")
#define CFENCE asm volatile("" ::: "memory")
#define START_BARRIER do { __builtin_amdgcn_s_barrier(); CFENCE; } while (0)

#define STAMP(slot) do { } while (0)

__device__ __forceinline__ auto mk_rsrc(const void* base, unsigned bytes) {
    return __builtin_amdgcn_make_buffer_rsrc(const_cast<void*>(base), 0, bytes, 0x00020000);
}
__device__ __forceinline__ void glds4(const char* gbase, unsigned voff, unsigned lds) {
    unsigned keep;
    asm volatile("s_mov_b32 %0, m0\n\ts_mov_b32 m0, %3\n\ts_nop 0\n\t"
                 "global_load_lds_dwordx4 %1, %2 sc0 sc1\n\t"
                 "global_load_lds_dwordx4 %1, %2 offset:1024 sc0 sc1\n\t"
                 "global_load_lds_dwordx4 %1, %2 offset:2048 sc0 sc1\n\t"
                 "global_load_lds_dwordx4 %1, %2 offset:3072 sc0 sc1\n\t"
                 "s_mov_b32 m0, %0"
                 : "=&s"(keep) : "v"(voff), "s"(gbase), "s"(lds) : "memory");
}
__device__ __forceinline__ void glds1(const char* gbase, unsigned voff, unsigned lds) {
    unsigned keep;
    asm volatile("s_mov_b32 %0, m0\n\ts_mov_b32 m0, %3\n\ts_nop 0\n\t"
                 "global_load_lds_dwordx4 %1, %2\n\t"
                 "s_mov_b32 m0, %0"
                 : "=&s"(keep) : "v"(voff), "s"(gbase), "s"(lds) : "memory");
}
__device__ __forceinline__ bool expired(u64 t0) { return __builtin_amdgcn_s_memrealtime() - t0 > DEADLINE; }
__device__ __forceinline__ unsigned lds_rd(volatile LAS unsigned* a) { return __builtin_amdgcn_readfirstlane(*a); }
__device__ __forceinline__ void lds_wait_ge(volatile LAS unsigned* a, unsigned v) {
    while ((int)(lds_rd(a) - v) < 0) __builtin_amdgcn_s_sleep(1);
    CFENCE;
}
__device__ __forceinline__ void lds_spin_ge(volatile LAS unsigned* a, unsigned v) {
    while ((int)(lds_rd(a) - v) < 0) { }
    CFENCE;
}
__device__ __forceinline__ void lds_add(volatile LAS unsigned* a) {
    __hip_atomic_fetch_add((LAS unsigned*)a, 1u, RLX, WGS);
}
__device__ __forceinline__ unsigned wave_max(unsigned v) {
    v = max(v, (unsigned)__builtin_amdgcn_update_dpp(0, (int)v, 0x111, 0xf, 0xf, false));
    v = max(v, (unsigned)__builtin_amdgcn_update_dpp(0, (int)v, 0x112, 0xf, 0xf, false));
    v = max(v, (unsigned)__builtin_amdgcn_update_dpp(0, (int)v, 0x114, 0xf, 0xf, false));
    v = max(v, (unsigned)__builtin_amdgcn_update_dpp(0, (int)v, 0x118, 0xf, 0xf, false));
    return max(max((unsigned)__builtin_amdgcn_readlane((int)v, 15), (unsigned)__builtin_amdgcn_readlane((int)v, 31)),
               max((unsigned)__builtin_amdgcn_readlane((int)v, 47), (unsigned)__builtin_amdgcn_readlane((int)v, 63)));
}
__device__ __forceinline__ unsigned lane_prefix(unsigned c, unsigned* total) {
    unsigned pre = 0, tot = 0;
#pragma unroll
    for (int k = 0; k < 6; ++k) {
        const u64 m = __ballot((c >> k) & 1u);
        pre += __builtin_amdgcn_mbcnt_hi((unsigned)(m >> 32), __builtin_amdgcn_mbcnt_lo((unsigned)m, 0u)) << k;
        tot += (unsigned)__builtin_popcountll(m) << k;
    }
    *total = tot;
    return pre;
}
__device__ __forceinline__ unsigned quant4(v4f y, float inv) {
    unsigned r[4];
#pragma unroll
    for (int e = 0; e < 4; ++e) r[e] = __float_as_uint(fminf(__builtin_fmaf(y[e], inv, 12582912.0f), 12582912.0f + 127.0f));
    return __builtin_amdgcn_perm(r[1], r[0], 0x0c0c0400u) | __builtin_amdgcn_perm(r[3], r[2], 0x04000c0cu);
}
__device__ __forceinline__ unsigned wait_granule(const char* addr, unsigned* tmo, unsigned seq, u64 t0) {
    for (;;) {
        u64 f;
        asm volatile("s_load_dwordx2 %0, %1, 0x0 glc\n\ts_waitcnt lgkmcnt(0)" : "=s"(f) : "s"(addr) : "memory");
        if ((unsigned)(f >> 32) == seq) return (unsigned)f;
        const u64 now = __builtin_amdgcn_s_memrealtime();
        if (now - t0 > 50000ull) {
            if (now - t0 > DEADLINE || __hip_atomic_load(tmo, RLX, AGT) == seq) {
                __hip_atomic_store(tmo, seq, RLX, AGT);
                return 0xffffffffu;
            }
        }
        __builtin_amdgcn_s_sleep(2);
    }
}

__global__ void __launch_bounds__(NWAVE * 64, 4) fused_gemm_ar_q8(Params p) {
    extern __shared__ __attribute__((aligned(16))) char smem[];
    const int tid = threadIdx.x;
    const int lane = tid & 63;
    const int wave = __builtin_amdgcn_readfirstlane(tid >> 6);
    const int b = blockIdx.x;
    const int rank = p.rank;
    char* const myws = p.ws[rank];
    unsigned* const tmo = (unsigned*)(myws + OFF_TMO);
    volatile LAS unsigned* aux = (volatile LAS unsigned*)(smem + LDS_AUX);
    const unsigned ldsbase = (unsigned)(size_t)(LAS char*)smem;
    const int xcd = b & 7, idx = b >> 3;
    const int tm = (xcd >> 1) * 8 + (idx & 7);
    const int tn = (xcd & 1) * 4 + (idx >> 3);
    const int fm = lane & 15, fq = lane >> 4;

    if (wave < 4) {
        if (tid < 64) aux[tid] = 0u;
        const int wm = wave >> 1, wn = wave & 1;
        v4f acc[2][4];
#pragma unroll
        for (int i = 0; i < 2; ++i)
#pragma unroll
            for (int j = 0; j < 4; ++j) acc[i][j] = v4f{0.f, 0.f, 0.f, 0.f};
        const unsigned xaddr = (unsigned)wm * 2048u + 16u * (unsigned)(4 * fm + (fq ^ ((4 - (fm >> 2)) & 3)));
        unsigned waddr;
        {
            const unsigned gg = (unsigned)fq, aa = (unsigned)((lane >> 2) & 3), bb = (unsigned)(lane & 3);
            const unsigned pp = bb >> 1, beta = bb & 1;
            const unsigned pos = ((2u * beta) ^ aa) | ((gg & 1u) << 2) | (pp << 3) | (aa << 4) | ((gg >> 1) << 6);
            waddr = 4096u + ((unsigned)wn * 2u + pp) * 2048u + 16u * pos + 8u * beta;
        }
        LGKM0;
        START_BARRIER;
        if (wave == 0) STAMP(0);
        {
            static_assert(NSLOT == 12 && NUNIT == 21 * 6 + 2, "ring unrolling");
            v8s xf[2][2], wf[2][4];
            unsigned xa = ldsbase + xaddr + 6u * SLOT, wa[8];
#pragma unroll
            for (int t = 0; t < 8; ++t)
                wa[t] = ldsbase + (waddr ^ (unsigned)(((t >> 2) & 1) << 4) ^ (unsigned)(((t >> 1) & 1) << 3) ^ (unsigned)((t & 1) << 7)) + 6u * SLOT;
            unsigned fa = ldsbase + LDS_AUX + (unsigned)(A_FULL + 6) * 4u;
            const unsigned consa = ldsbase + LDS_AUX + (unsigned)(A_CONS + wave) * 4u;
            auto rd = [&](int set, int i) {
#pragma unroll
                for (int t = 0; t < 4; ++t) {
                    v4s lo = __builtin_amdgcn_ds_read_tr16_b64_v4i16((LAS v4s*)(size_t)(wa[2 * t] + (unsigned)i * SLOT));
                    v4s hi = __builtin_amdgcn_ds_read_tr16_b64_v4i16((LAS v4s*)(size_t)(wa[2 * t + 1] + (unsigned)i * SLOT));
                    wf[set][t] = __builtin_shufflevector(lo, hi, 0, 1, 2, 3, 4, 5, 6, 7);
                    if (t == 0) {
#pragma unroll
                        for (int mt = 0; mt < 2; ++mt) xf[set][mt] = *(const LAS v8s*)(size_t)(xa + (unsigned)i * SLOT + mt * 1024);
                    }
                }
            };
            auto mm = [&](int set) {
#pragma unroll
                for (int t = 0; t < 4; ++t)
#pragma unroll
                    for (int mt = 0; mt < 2; ++mt)
                        acc[mt][t] = __builtin_amdgcn_mfma_f32_16x16x32_bf16((v8bf)wf[set][t], (v8bf)xf[set][mt], acc[mt][t], 0, 0, 0);
            };
            auto step = [&](int n, int i) {
                LGKM0;
                *(volatile LAS unsigned*)(size_t)consa = (unsigned)(n + 1);
                const unsigned f = *(volatile LAS unsigned*)(size_t)(fa + (unsigned)i * 4u);
                CFENCE;
                rd((i & 1) ^ 1, i);
                mm(i & 1);
                __builtin_amdgcn_sched_group_barrier(0x100, 2, 0); __builtin_amdgcn_sched_group_barrier(0x8, 1, 0);
                __builtin_amdgcn_sched_group_barrier(0x100, 2, 0); __builtin_amdgcn_sched_group_barrier(0x8, 1, 0);
                __builtin_amdgcn_sched_group_barrier(0x100, 2, 0); __builtin_amdgcn_sched_group_barrier(0x8, 1, 0);
                __builtin_amdgcn_sched_group_barrier(0x100, 2, 0); __builtin_amdgcn_sched_group_barrier(0x8, 1, 0);
                __builtin_amdgcn_sched_group_barrier(0x100, 2, 0); __builtin_amdgcn_sched_group_barrier(0x8, 4, 0);
                __builtin_amdgcn_sched_barrier(0);
                if ((int)(__builtin_amdgcn_readfirstlane(f) - (unsigned)(n + 2)) < 0) {
                    lds_wait_ge((volatile LAS unsigned*)(size_t)(fa + (unsigned)i * 4u), (unsigned)(n + 2));
                    rd((i & 1) ^ 1, i);
                }
            };
            __builtin_amdgcn_s_setprio(2);
            lds_wait_ge(aux + A_FULL + 11, 1u);
            rd(0, 5);
            unsigned dl = 0u - 6u * SLOT, df = 0u - 24u;
            auto flip = [&]() {
                xa += dl;
#pragma unroll
                for (int t = 0; t < 8; ++t) wa[t] += dl;
                fa += df;
                dl = 0u - dl;
                df = 0u - df;
                asm volatile("" : "+v"(xa), "+v"(wa[0]), "+v"(wa[1]), "+v"(wa[2]), "+v"(wa[3]), "+v"(wa[4]), "+v"(wa[5]), "+v"(wa[6]), "+v"(wa[7]), "+v"(fa));
            };
            for (int n = 0; n < NUNIT - 2; n += 6) {
                flip();
#pragma unroll
                for (int i = 0; i < 6; ++i) step(n + i, i);
            }
            flip();
            step(NUNIT - 2, 0);
            mm(1);
        }
        if (wave == 0) STAMP(3);
        unsigned m32 = 0;
#pragma unroll
        for (int i = 31; i >= 0; --i) m32 = m32 + m32 + (acc[i >> 4][(i >> 2) & 3][i & 3] > 0.f ? 1u : 0u);
        *(LAS unsigned*)(smem + L_MASK + (unsigned)tid * 4u) = m32;
        CFENCE;
        if (lane == 0) lds_add(aux + A_CNTM);
        __builtin_amdgcn_sched_barrier(0);
        float mx = 0.f;
#pragma unroll
        for (int i = 0; i < 2; ++i)
#pragma unroll
            for (int j = 0; j < 4; ++j)
#pragma unroll
                for (int e = 0; e < 4; ++e) {
                    acc[i][j][e] = fmaxf(acc[i][j][e], 0.f);
                    mx = fmaxf(mx, acc[i][j][e]);
                }
        const unsigned wmx = wave_max(__float_as_uint(mx));
        if (lane == 0) {
            __hip_atomic_fetch_max((LAS unsigned*)(aux + A_MAX), wmx, RLX, WGS);
            CFENCE;
            lds_add(aux + A_CNT);
        }
        unsigned tot;
        const unsigned pre = lane_prefix((unsigned)__builtin_popcount(m32), &tot);
        if (lane == 0) { aux[A_TOT + wave] = tot; CFENCE; lds_add(aux + A_CNT1); }
        lds_wait_ge(aux + A_CNT1, 4u);
        const v4u tt = *(const volatile LAS v4u*)(aux + A_TOT);
        const unsigned T = __builtin_amdgcn_readfirstlane(tt.x + tt.y + tt.z + tt.w);
        {
            unsigned a = L_STREAM + 4u * (pre + (wave > 0 ? tt.x : 0u) + (wave > 1 ? tt.y : 0u) + (wave > 2 ? tt.z : 0u));
            const unsigned dummy = L_DUMMY + (unsigned)tid * 4u;
#pragma unroll
            for (int i = 0; i < 2; ++i)
#pragma unroll
                for (int j = 0; j < 4; ++j)
#pragma unroll
                    for (int e = 0; e < 4; ++e) {
                        const bool on = acc[i][j][e] > 0.f;
                        *(LAS float*)(smem + (on ? a : dummy)) = acc[i][j][e];
                        a += on ? 4u : 0u;
                    }
        }
        CFENCE;
        if (lane == 0) lds_add(aux + A_CNT2);
        lds_wait_ge(aux + A_CNT2, 4u);
        v4f sv[2][4];
#pragma unroll
        for (int h = 0; h < 2; ++h)
#pragma unroll
            for (int j = 0; j < 4; ++j)
                sv[h][j] = *(const LAS v4f*)(smem + L_STREAM + (unsigned)((wave + 4 * h) * 4096 + lane * 64 + j * 16));
        lds_spin_ge(aux + A_GO, 1u);
        if (wave == 0) STAMP(4);
        const float amax = __uint_as_float(aux[A_GM]);
        const bool dead = aux[A_DEAD] != 0u;
        const float inv = amax > 0.f ? 127.0f / amax : 0.f;
        const v4u bad = {0x80808080u, 0x80808080u, 0x80808080u, 0x80808080u};
#pragma unroll
        for (int h = 0; h < 2; ++h) {
            if (h == 0 || T > 4096u) {
                const unsigned c = (unsigned)((wave + 4 * h) * 64 + lane);
                const unsigned q0 = quant4(sv[h][0], inv), q1 = quant4(sv[h][1], inv), q2 = quant4(sv[h][2], inv), q3 = quant4(sv[h][3], inv);
                LAS unsigned* const dst = (LAS unsigned*)(smem + L_PACK + c * 12u);
                if (c * 16u < T) {
                    dst[0] = (q0 & 0x3f3f3f3fu) | ((q3 << 6) & 0xc0c0c0c0u);
                    dst[1] = (q1 & 0x3f3f3f3fu) | ((q3 << 4) & 0xc0c0c0c0u);
                    dst[2] = (q2 & 0x3f3f3f3fu) | ((q3 << 2) & 0xc0c0c0c0u);
                    unsigned m = 0;
                    if ((q0 | q1 | q2 | q3) & 0x40404040u) {
                        m = ((((q0 >> 6) & 0x01010101u) * 0x10204080u) >> 28) | (((((q1 >> 6) & 0x01010101u) * 0x10204080u) >> 28) << 4)
                          | (((((q2 >> 6) & 0x01010101u) * 0x10204080u) >> 28) << 8) | (((((q3 >> 6) & 0x01010101u) * 0x10204080u) >> 28) << 12);
                        unsigned e = __hip_atomic_fetch_add((LAS unsigned*)(aux + A_EXC), (unsigned)__builtin_popcount(m), RLX, WGS);
                        for (unsigned r = m; r && e < 512u; r &= r - 1u, ++e)
                            *(LAS unsigned short*)(smem + L_LIST + 2u * e) = (unsigned short)(c * 16u + (unsigned)__builtin_ctz(r));
                    }
                    *(LAS unsigned short*)(smem + L_PLANE + 2u * c) = (unsigned short)m;
                }
            }
        }
        CFENCE;
        if (lane == 0) lds_add(aux + A_CNT3);
        auto ro = mk_rsrc(p.out[rank], (unsigned)NCOL * 4096u);
        const unsigned ooff = (unsigned)((rank >> 2) * 2048 + tm * 64 + wm * 32 + fm) * NCOL + (unsigned)((rank & 3) * 1024 + tn * 128 + wn * 64 + fq * 16);
#pragma unroll
        for (int mt = 0; mt < 2; ++mt) {
            const unsigned d0 = quant4(acc[mt][0], inv), d1 = quant4(acc[mt][1], inv), d2 = quant4(acc[mt][2], inv), d3 = quant4(acc[mt][3], inv);
            const bool odd = fq & 1;
            v4u qv = {odd ? d1 : d0, odd ? d0 : d1, odd ? d3 : d2, odd ? d2 : d3};
            if (dead) qv = bad;
            __builtin_amdgcn_raw_buffer_store_b128(qv, ro, ooff + mt * 16 * NCOL, 0, 16);
        }
        return;
    }

    unsigned* const ctr = (unsigned*)(myws + OFF_CTR) + (b * 16 + wave);
    const unsigned seqv = __hip_atomic_load(ctr, RLX, AGT) + 1u;
    const int q = b & 31;
    const int dest = (rank + (b >> 5)) & 7;

    unsigned* const prog = (unsigned*)(p.ws[dest] + OFF_PROG + (unsigned)((lane & (FREP - 1)) * 32 + q) * 128u + (unsigned)rank * 4u);

    const unsigned xo = (unsigned)((dest >> 2) * 2048 + q * 64 + (lane >> 2)) * (KC * 2) + (unsigned)((lane & 3) ^ ((4 - (lane >> 4)) & 3)) * 16u;
    const unsigned wo = (unsigned)((lane >> 4) | ((((lane >> 3) & 1) ^ (q & 1)) << 2) | (((lane >> 2) & 1) << 3)) * (NCOL * 2)
                      + (unsigned)((dest & 3) * 1024 + q * 32) * 2u + (unsigned)((lane & 3) ^ (lane >> 4)) * 16u;
    int st;
    unsigned seq;

    if (wave == W_COPY || wave == W_COPY + 1) {
        st = wave - W_COPY;
        __builtin_amdgcn_s_setprio(3);
        v4u d[3];
        d[0] = *(const v4u*)(p.x + xo + (unsigned)(st * 16 * (KC * 2)));
        d[1] = *(const v4u*)(p.x + xo + (unsigned)((st + 2) * 16 * (KC * 2)));
        d[2] = *(const v4u*)(p.w + wo + (unsigned)(st * 16 * (NCOL * 2)));
        if (st == 0) STAMP(8);
        auto rs = mk_rsrc(p.ws[dest] + OFF_STAGE + (size_t)rank * NSUB * UNIT, NSUB * UNIT);
        const unsigned ox = (unsigned)q * 4096u + (unsigned)(st * 1024 + lane * 16);
        const unsigned ow = UNIT_X + (unsigned)q * 2048u + (unsigned)(st * 1024 + lane * 16);
        const LAS char* const src = (const LAS char*)smem + LDS_CPY + (unsigned)(st * 1024 + lane * 16);
        const unsigned pace = dest != rank ? PACE : 0u;
        VMCNT0_SEEN;
        const unsigned tc = (unsigned)__builtin_amdgcn_s_memrealtime() - (xcd == 0 ? XADV : 0u);
        int grp = 0;
        for (int s = 0; s < NSUB; ++s) {
            if (s > 0) {
                lds_wait_ge(aux + A_FETCHED, (unsigned)s);
#pragma unroll
                for (int j = 0; j < 3; ++j) d[j] = *(const LAS v4u*)(src + grp * 6144 + j * 2048);
                grp = grp == CG - 1 ? 0 : grp + 1;
                LGKM0;
                if (lane == 0) aux[A_TAKEN + st] = (unsigned)s;
            }
#pragma unroll
            for (int j = 0; j < 3; ++j) {
                VMCNT(1);
                if (j == 1 && s > 0 && lane == 0) aux[A_SENT + st] = (unsigned)s;
                const unsigned due = (unsigned)(s * 6 + j * 2 + st) * pace / 6u;
                while ((unsigned)__builtin_amdgcn_s_memrealtime() - tc < due) __builtin_amdgcn_s_sleep(1);
                __builtin_amdgcn_raw_buffer_store_b128(d[j], rs, (j < 2 ? ox + j * 2048 : ow) + (unsigned)s * UNIT, 0, 17);
                if (s == 0 && j == 0) { if (st == 0) STAMP(9); START_BARRIER; }
            }
        }
        VMCNT(0);
        seq = __builtin_amdgcn_readfirstlane(seqv);
        unsigned old = 0;
        if (lane == 0) old = __hip_atomic_fetch_add((LAS unsigned*)(aux + A_CDONE), 1u, RLX, WGS);
        if (__builtin_amdgcn_readfirstlane(old) == 1u && lane < FREP) __hip_atomic_fetch_max(prog, seq * 32u + (unsigned)NSUB, RLX, SYS);
        if (lane == 0) __hip_atomic_store(ctr, seq, RLX, AGT);
        STAMP(10);
    } else {

    if (wave == W_FETCH) {
        auto fetch = [&](int g) {
            const unsigned l0 = ldsbase + LDS_CPY + (unsigned)(g % CG) * 6144u;
#pragma unroll
            for (int j = 0; j < 4; ++j) glds1(p.x, xo + (unsigned)((g + 1) * 64 + j * 16 * (KC * 2)), l0 + j * 1024u);
#pragma unroll
            for (int j = 0; j < 2; ++j) glds1(p.w, wo + (unsigned)(((g + 1) * 32 + j * 16) * (NCOL * 2)), l0 + 4096u + j * 1024u);
        };
        constexpr int NG = NSUB - 1;
#pragma unroll
        for (int g = 0; g < CG; ++g) fetch(g);
        START_BARRIER;
#pragma unroll
        for (int g = 0; g < NG; ++g) {
            VMCNTN(6 * ((CG + g < NG ? CG + g : NG) - g - 1));
            if (lane == 0) aux[A_FETCHED] = (unsigned)(g + 1);
            if (g + CG < NG) { lds_wait_ge(aux + A_TAKEN, (unsigned)(g + 1)); lds_wait_ge(aux + A_TAKEN + 1, (unsigned)(g + 1)); fetch(g + CG); }
        }
        VMCNT0_SEEN;
        seq = __builtin_amdgcn_readfirstlane(seqv);
        if (lane == 0) __hip_atomic_store(ctr, seq, RLX, AGT);
        const int dst = (rank + 1 + ((lane < 7 ? lane : 0) + b % 7) % 7) & 7;
        u64* const fl = (u64*)(p.ws[dst] + OFF_P2F + (unsigned)b * 128u + (unsigned)rank * 8u);
        lds_wait_ge(aux + A_CNT3, 4u);
        const u64 fv = ((u64)seq << 32) | (aux[A_DEAD] ? 0xffffffffu : aux[A_EXC]);
        int done = 0;
        while (done < 6) {
            int d2;
            for (;;) {
                const v4u pp = *(const volatile LAS v4u*)(aux + A_P2);
                d2 = (int)__builtin_amdgcn_readfirstlane(min(pp.x, pp.y));
                if (d2 > done) break;
                __builtin_amdgcn_s_sleep(1);
            }
            if (lane >= done && lane < d2 && lane < 6) __hip_atomic_store(fl, fv, RLX, SYS);
            done = d2;
        }
        return;
    }

    seq = __builtin_amdgcn_readfirstlane(seqv);
    if (lane == 0) __hip_atomic_store(ctr, seq, RLX, AGT);
    const u64 t0 = __builtin_amdgcn_s_memrealtime();
    VMCNT0_SEEN;
    START_BARRIER;

    if (wave == W_POLL) {
        const char* px = myws + OFF_PROG + (unsigned)tm * 128u;
        const char* pw = myws + OFF_PROG + (unsigned)tn * 512u;
        const unsigned r8 = (unsigned)(((idx & 7) ^ (((idx >> 3) << 1) | (xcd & 1))) & (FREP - 1));
        unsigned radd = (r8 & (PREP - 1)) * 4096u, radd2 = (r8 & ~(PREP - 1)) * 4096u;
        int pvk = 0;
        const unsigned pbase = seq * 32u;
        unsigned sent = 0;
        bool stop = false, fin = false, fast = false;
        while (!fin) {
            const u64 now = __builtin_amdgcn_s_memrealtime();
            u64 gap = 0;
            if (!stop) {
                int pv = 0, lo = NSUB;
                v8u fx, f0, f1, f2, f3;
                asm volatile("s_load_dwordx8 %0, %5, 0x0 glc\n\ts_load_dwordx8 %1, %6, 0x0 glc\n\ts_load_dwordx8 %2, %6, 0x80 glc\n\t"
                             "s_load_dwordx8 %3, %6, 0x100 glc\n\ts_load_dwordx8 %4, %6, 0x180 glc\n\ts_waitcnt lgkmcnt(0)"
                             : "=&s"(fx), "=&s"(f0), "=&s"(f1), "=&s"(f2), "=&s"(f3) : "s"(px), "s"(pw) : "memory");
#pragma unroll
                for (int c = 0; c < 8; ++c) {
                    const int m = min(min(min((int)(f0[c] - pbase), (int)(f1[c] - pbase)), min((int)(f2[c] - pbase), (int)(f3[c] - pbase))), (int)(fx[c] - pbase));
                    lo = min(lo, m);
                    pv = lane == c ? m : pv;
                }
                pv = pvk = max(pv, pvk);
                if (lo >= (int)FINE_FROM) { px += radd; pw += radd; radd = 0; fast = true; }
                if (lo >= NSUB - 1) { px += radd2; pw += radd2; radd2 = 0; }
                stop = lo >= NSUB;
                if (!stop && now - t0 > 50000ull) {
                    if (now - t0 > DEADLINE || __hip_atomic_load(tmo, RLX, AGT) == seq) {
                        __hip_atomic_store(tmo, seq, RLX, AGT);
                        pv = NSUB;
                        stop = true;
                        if (lane == 0) aux[A_DEAD] = 1u;
                    }
                }
                if (lane < 8) aux[A_PRG + lane] = (unsigned)pv;
                if (stop) STAMP(15);
                gap = fast ? GAP_FAST : GAP_SLOW;
            }
            do {
                if (lds_rd(aux + A_CDONE) == 2u) { if (stop) { fin = true; break; } }
                else {
                    const unsigned d = min(lds_rd(aux + A_SENT), lds_rd(aux + A_SENT + 1));
                    if (d >= sent + (d > FINE_FROM ? 1u : FSTEP)) {
                        sent = d;
                        if (lane < (d > FINE_FROM ? PREP : 1)) __hip_atomic_fetch_max(prog, pbase + d, RLX, SYS);
                    }
                }
                __builtin_amdgcn_s_sleep(1);
            } while (__builtin_amdgcn_s_memrealtime() - now < gap);
        }
        return;
    } else if (wave == W_SCHED) {
        const int csrc = (rank + lane) & 7;
        int nxt = lane < 8 ? 0 : NSUB;
        int asg = 0;
        while (asg < NUNIT) {
            const int prg = (int)aux[A_PRG + csrc];
            unsigned av = (unsigned)__ballot(prg > nxt);
            if (av == 0) { __builtin_amdgcn_s_sleep(1); continue; }
            while (av) {
                const int k = __builtin_popcount(av & ((1u << (lane & 7)) - 1u));
                if (prg > nxt) { aux[A_DESC + asg + k] = (unsigned)(csrc * NSUB + nxt) * UNIT; ++nxt; }
                asg += __builtin_popcount(av);
                CFENCE;
                if (lane == 0) aux[A_ASG] = (unsigned)asg;
                av = (unsigned)__ballot(prg > nxt);
            }
        }
        STAMP(23);
        lds_wait_ge(aux + A_CNT, 4u);
        if (lane == 0) __hip_atomic_store((u64*)(myws + OFF_WGMAX + (unsigned)b * 8u), ((u64)seq << 32) | aux[A_MAX], RLX, AGT);
        bool dead = __hip_atomic_load(tmo, RLX, AGT) == seq;
        if (b == 0) {
            auto rg = mk_rsrc(myws + OFF_WGMAX, NWG * 8);
            unsigned mx;
            for (;;) {
                v4u g0 = __builtin_amdgcn_raw_buffer_load_b128(rg, (unsigned)lane * 32u, 0, 17);
                v4u g1 = __builtin_amdgcn_raw_buffer_load_b128(rg, (unsigned)lane * 32u + 16u, 0, 17);
                unsigned tm_ = __hip_atomic_load(tmo, RLX, AGT);
                bool ok = g0.y == seq && g0.w == seq && g1.y == seq && g1.w == seq;
                mx = max(max(g0.x, g0.z), max(g1.x, g1.z));
                if (__all(ok)) break;
                if (tm_ == seq || expired(t0)) { __hip_atomic_store(tmo, seq, RLX, AGT); dead = true; break; }
            }
            mx = wave_max(mx);
            char* dst = p.ws[(lane >> 2) & 7] + OFF_RMAX + (unsigned)(lane & 3) * 512u + (unsigned)rank * 8u;
            if (lane < 32) __hip_atomic_store((u64*)dst, ((u64)seq << 32) | mx, RLX, SYS);
            if (lane == 0) aux[A_LEAD] = 1u;
            STAMP(7);
            const char* const ra = myws + OFF_RMAX;
            unsigned seen = 0, flip = 0;
            mx = 0;
            for (;;) {
                v16u g0, g1;
                asm volatile("s_load_dwordx16 %0, %2, 0x0 glc\n\ts_load_dwordx16 %1, %2, 0x200 glc\n\ts_waitcnt lgkmcnt(0)"
                             : "=&s"(g0), "=&s"(g1) : "s"(ra + flip) : "memory");
                flip ^= 1024u;
#pragma unroll
                for (int c = 0; c < 8; ++c) {
                    if (g0[2 * c + 1] == seq) { seen |= 1u << c; mx = max(mx, g0[2 * c]); }
                    if (g1[2 * c + 1] == seq) { seen |= 1u << c; mx = max(mx, g1[2 * c]); }
                }
                if (seen == 255u) break;
                const u64 now = __builtin_amdgcn_s_memrealtime();
                if (now - t0 > 50000ull) {
                    if (now - t0 > DEADLINE || __hip_atomic_load(tmo, RLX, AGT) == seq) { __hip_atomic_store(tmo, seq, RLX, AGT); dead = true; break; }
                }
            }
            if (lane < 32) __builtin_amdgcn_raw_buffer_store_b128(v4u{mx, seq, (unsigned)__builtin_amdgcn_s_memrealtime(), seq}, mk_rsrc(myws + OFF_GMAX, 4096u), (unsigned)lane * 128u, 0, 16);
            STAMP(14);
        }
        const char* const ga = myws + OFF_GMAX + (unsigned)(b & 31) * 128u;
        unsigned gm, tg;
        for (;;) {
            v4u g;
            asm volatile("s_load_dwordx4 %0, %1, 0x0 glc\n\ts_waitcnt lgkmcnt(0)" : "=s"(g) : "s"(ga) : "memory");
            gm = g.x;
            tg = g.z;
            if (g.y == seq && g.w == seq) break;
            const u64 now = __builtin_amdgcn_s_memrealtime();
            if (now - t0 > 50000ull) {
                if (now - t0 > DEADLINE || __hip_atomic_load(tmo, RLX, AGT) == seq) { __hip_atomic_store(tmo, seq, RLX, AGT); dead = true; break; }
            }
        }
        if (dead) tg = (unsigned)__builtin_amdgcn_s_memrealtime();
        if (lane == 0) {
            aux[A_GM] = gm;
            aux[A_TGO] = tg + TGO_ADD;
            if (dead) aux[A_DEAD] = 1u;
            CFENCE;
            aux[A_GO] = 1u;
        }
        return;
    } else {
        const int l = wave - 4;
        const char* const stage = myws + OFF_STAGE;
        const unsigned vx = (unsigned)tm * 4096u + (unsigned)lane * 16u;
        const unsigned vw = UNIT_X + (unsigned)tn * 8192u + (unsigned)lane * 16u;
        auto issue = [&](int n, int sl) {
            const char* gb = stage + lds_rd(aux + A_DESC + n);
            const unsigned ld = ldsbase + (unsigned)sl * SLOT;
            glds4(gb, vx, ld);
            glds4(gb, vw, ld + 4096u);
            glds4(gb + 4096, vw, ld + 8192u);
        };
        auto ready = [&](int n) {
            const unsigned as = aux[A_ASG];
            const v4u cs = *(const volatile LAS v4u*)(aux + A_CONS);
            const bool r = __builtin_amdgcn_readfirstlane(min(as, min(min(cs.x, cs.y), min(cs.z, cs.w)) + NSLOT)) > (unsigned)n;
            CFENCE;
            return r;
        };
        int slot = l == 0 ? NSLOT - 1 : l - 1;
        while (!ready(l)) __builtin_amdgcn_s_sleep(1);
        issue(l, slot);
        for (int n = l;;) {
            const int nn = n + NL, ns = slot + NL >= NSLOT ? slot + NL - NSLOT : slot + NL;
            const bool next = nn < NUNIT && ready(nn);
            if (next) { issue(nn, ns); VMCNT(12); } else VMCNT(0);
            if (lane == 0) aux[A_FULL + slot] = (unsigned)(n + 1);
            if (nn >= NUNIT) break;
            if (!next) { while (!ready(nn)) __builtin_amdgcn_s_sleep(1); issue(nn, ns); }
            n = nn;
            slot = ns;
        }
        const int src = (rank + 1 + l) & 7;
        const char* const fl = myws + OFF_P2F + (unsigned)b * 128u + (unsigned)src * 8u;
        auto rp = mk_rsrc(myws + OFF_P2 + (unsigned)(src * NWG + b) * P2_PITCH, P2_PITCH);
        lds_wait_ge(aux + A_GO, 1u);
        {
            const unsigned tg = lds_rd(aux + A_TGO);
            while ((int)((unsigned)__builtin_amdgcn_s_memrealtime() - tg - MWAIT) < 0) __builtin_amdgcn_s_sleep(2);
        }
        unsigned mk[4];
#pragma unroll
        for (int w = 0; w < 4; ++w) mk[w] = __builtin_amdgcn_raw_buffer_load_b32(rp, (unsigned)(w * 256 + lane * 4), 0, 17);
        unsigned nexc = 0;
        unsigned sel[8][6], off[8];
        v3u pk[8];
        unsigned ex0, xi, nch;
        int nit;
        for (int pass = 0;; ++pass) {
            unsigned base = 0;
#pragma unroll
            for (int w = 0; w < 4; ++w) {
                unsigned tot;
                const unsigned pre = lane_prefix((unsigned)__builtin_popcount(mk[w]), &tot);
                off[2 * w] = base + pre;
                off[2 * w + 1] = base + pre + (unsigned)__builtin_popcount(mk[w] & 0xffffu);
                base += tot;
#pragma unroll
                for (int mt = 0; mt < 2; ++mt) {
                    const unsigned m16 = (mk[w] >> (16 * mt)) & 0xffffu;
#pragma unroll
                    for (int t = 0; t < 4; ++t) {
                        const unsigned bits = (((m16 >> (4 * t)) & 15u) * 0x00204081u) & 0x01010101u;
                        const unsigned ix = bits * 0x01010100u + (unsigned)__builtin_popcount(m16 & ((1u << (4 * t)) - 1u)) * 0x01010101u;
                        if (t < 2) {
                            const unsigned ma = bits * 0xffu;
                            sel[2 * w + mt][t] = (ix & ma) | (0x0c0c0c0cu & ~ma);
                        } else {
                            const unsigned hi = (ix >> 3) & bits;
                            const unsigned ma = (bits ^ hi) * 0xffu, mb = hi * 0xffu;
                            sel[2 * w + mt][t] = (ix & ma) | (0x0c0c0c0cu & ~ma);
                            sel[2 * w + mt][t + 2] = (ix & 0x07070707u & mb) | (0x0c0c0c0cu & ~mb);
                        }
                    }
                }
            }
#pragma unroll
            for (int i = 0; i < 8; ++i)
                asm volatile("" : "+v"(sel[i][0]), "+v"(sel[i][1]), "+v"(sel[i][2]), "+v"(sel[i][3]), "+v"(sel[i][4]), "+v"(sel[i][5]), "+v"(off[i]));
            if (pass == 0) {
                if (l == 0) STAMP(12);
                nexc = wait_granule(fl, tmo, seq, t0);
                if (l == 0) STAMP(13);
            }
            nit = (int)((base + 1023u) >> 10);
#pragma unroll
            for (int i = 0; i < 8; ++i)
                if (i < nit) pk[i] = __builtin_amdgcn_raw_buffer_load_b96(rp, 1024u + (unsigned)(i * 64 + lane) * 12u, 0, 17);
            nch = (base + 15u) >> 4;
            ex0 = 1024u + ((nch * 12u + 15u) & ~15u);
            xi = __builtin_amdgcn_raw_buffer_load_b16(rp, ex0 + 2u * (unsigned)lane, 0, 17);
            if (pass > 0) break;
            bool same = true;
#pragma unroll
            for (int w = 0; w < 4; ++w) {
                const unsigned m2 = __builtin_amdgcn_raw_buffer_load_b32(rp, (unsigned)(w * 256 + lane * 4), 0, 17);
                same = same && m2 == mk[w];
                mk[w] = m2;
            }
            if (__all(same)) break;
        }
        const bool ok = nexc != 0xffffffffu;
        LAS char* const dec = (LAS char*)smem + L_DEC + (unsigned)l * DEC_PITCH;
        {
            const bool plane = ok && nexc > nch;
#pragma unroll
            for (int i = 0; i < 8; ++i) {
                if (i < nit) {
                    v4u q = {pk[i].x & 0x3f3f3f3fu, pk[i].y & 0x3f3f3f3fu, pk[i].z & 0x3f3f3f3fu,
                             ((pk[i].x >> 6) & 0x03030303u) | ((pk[i].y >> 4) & 0x0c0c0c0cu) | ((pk[i].z >> 2) & 0x30303030u)};
                    if (plane) {
                        const unsigned m = __builtin_amdgcn_raw_buffer_load_b16(rp, ex0 + (unsigned)(i * 64 + lane) * 2u, 0, 17);
#pragma unroll
                        for (int t = 0; t < 4; ++t) q[t] |= ((((m >> (4 * t)) & 15u) * 0x00204081u) & 0x01010101u) << 6;
                    }
                    *(LAS v4u*)(dec + (unsigned)(i * 1024 + lane * 16)) = q;
                }
            }
            CFENCE;
            for (unsigned e = (unsigned)lane; ok && !plane && e < nexc; e += 64u) {
                if (e >= 64u) xi = __builtin_amdgcn_raw_buffer_load_b16(rp, ex0 + 2u * e, 0, 17);
                __hip_atomic_fetch_or((LAS unsigned*)(dec + (xi & 0x1ffcu)), 0x40u << (8u * (xi & 3u)), RLX, WGS);
            }
            CFENCE;
        }
        auto ro = mk_rsrc(p.out[rank], (unsigned)NCOL * 4096u);
        const unsigned ooff = (unsigned)((src >> 2) * 2048 + tm * 64 + fm) * NCOL + (unsigned)((src & 3) * 1024 + tn * 128 + fq * 16);
        const bool odd = fq & 1;
#pragma unroll
        for (int s = 0; s < 8; ++s) {
            const LAS unsigned* const wp = (const LAS unsigned*)(dec + (off[s] & ~3u));
            const unsigned r0 = wp[0], r1 = wp[1], r2 = wp[2], r3 = wp[3], r4 = wp[4];
            const unsigned w0 = __builtin_amdgcn_alignbyte(r1, r0, off[s]), w1 = __builtin_amdgcn_alignbyte(r2, r1, off[s]);
            const unsigned w2 = __builtin_amdgcn_alignbyte(r3, r2, off[s]), w3 = __builtin_amdgcn_alignbyte(r4, r3, off[s]);
            const unsigned d0 = __builtin_amdgcn_perm(w1, w0, sel[s][0]);
            const unsigned d1 = __builtin_amdgcn_perm(w1, w0, sel[s][1]);
            const unsigned d2 = __builtin_amdgcn_perm(w1, w0, sel[s][2]) | __builtin_amdgcn_perm(w3, w2, sel[s][4]);
            const unsigned d3 = __builtin_amdgcn_perm(w1, w0, sel[s][3]) | __builtin_amdgcn_perm(w3, w2, sel[s][5]);
            v4u qv = {odd ? d1 : d0, odd ? d0 : d1, odd ? d3 : d2, odd ? d2 : d3};
            if (!ok) qv = v4u{0x80808080u, 0x80808080u, 0x80808080u, 0x80808080u};
            __builtin_amdgcn_raw_buffer_store_b128(qv, ro, ooff + (unsigned)(((s >> 2) * 32 + (s & 1) * 16) * NCOL + ((s >> 1) & 1) * 64), 0, 16);
        }
        STAMP(16 + l);
        return;
    }
    }

    const unsigned poff = OFF_P2 + (unsigned)(rank * NWG + b) * P2_PITCH;
    const int rot = b % 7;
    lds_wait_ge(aux + A_ASG, (unsigned)NUNIT);
    lds_spin_ge(aux + A_CNTM, 4u);
    const unsigned tm0 = (unsigned)__builtin_amdgcn_s_memrealtime();
    {
        const v4u mk = *(const LAS v4u*)(smem + L_MASK + (unsigned)lane * 16u);
        if (b == 0) lds_wait_ge(aux + A_LEAD, 1u);
        for (int k = st; k < 7; k += 2) {
            const int dst = (rank + 1 + (k + rot) % 7) & 7;
            VMCNT(1);
            if (b != 0) while ((unsigned)__builtin_amdgcn_s_memrealtime() - tm0 < (unsigned)k * MPACE) __builtin_amdgcn_s_sleep(1);
            __builtin_amdgcn_raw_buffer_store_b128(mk, mk_rsrc(p.ws[dst] + poff, 1024u), (unsigned)lane * 16u, 0, 17);
        }
    }
    if (st == 0) STAMP(11);
    int last = 0, pub = 0;
    lds_spin_ge(aux + A_CNT3, 4u);
    CFENCE;
    const v4u tt = *(const volatile LAS v4u*)(aux + A_TOT);
    const unsigned nch = __builtin_amdgcn_readfirstlane((tt.x + tt.y + tt.z + tt.w + 15u) >> 4), nex = lds_rd(aux + A_EXC);
    const unsigned Vp = (nch * 12u + 15u) & ~15u;
    const unsigned xadd = (nex <= nch ? L_LIST : L_PLANE) - L_PACK - Vp;
    const unsigned Tr = (Vp + 2u * min(nex, nch) + 63u) & ~63u;
    const int np = (int)((Tr + 1023u) >> 10);
    const unsigned ps = np ? (((Tr >> 6) + (unsigned)np - 1u) / (unsigned)np) << 6 : 0u;
    unsigned tv = lds_rd(aux + A_TGO);
    if ((int)(tm0 + 7u * MPACE - tv) > 0) tv = tm0 + 7u * MPACE;
    for (int k = 0, j = st; ; j += 2) {
        while (j >= np && k < 7) { j -= np; ++k; }
        if (k >= 7) break;
        const unsigned o = (unsigned)j * ps;
        const unsigned ol = o + (unsigned)lane * 16u;
        const v4u pv = *(const LAS v4u*)(smem + L_PACK + ol + (ol >= Vp ? xadd : 0u));
        char* const dp = p.ws[(rank + 1 + (k + rot) % 7) & 7] + poff + 1024u + o;
        VMCNT(1);
        if (last > pub) {
            if (lane == 0) aux[A_P2 + st] = (unsigned)last;
            pub = last;
        }
        const unsigned due = (((unsigned)k * Tr + o) * VPACE) >> 10;
        while ((int)((unsigned)__builtin_amdgcn_s_memrealtime() - tv - due) < 0) __builtin_amdgcn_s_sleep(1);
        __builtin_amdgcn_raw_buffer_store_b128(pv, mk_rsrc(dp, o < Tr ? min(ps, Tr - o) : 0u), (unsigned)lane * 16u, 0, 17);
        last = k;
    }
    VMCNT(0);
    {
        unsigned old = 0;
        if (lane == 0) old = __hip_atomic_fetch_add((LAS unsigned*)(aux + A_PDONE), 1u, RLX, WGS);
        if (__builtin_amdgcn_readfirstlane(old) == 1u && lane == 0)
            __hip_atomic_store((u64*)(p.ws[(rank + 1 + (6 + rot) % 7) & 7] + OFF_P2F + (unsigned)b * 128u + (unsigned)rank * 8u),
                               ((u64)seq << 32) | (aux[A_DEAD] ? 0xffffffffu : nex), RLX, SYS);
    }
    if (lane == 0) aux[A_P2 + st] = 7u;
    STAMP(5);
}

}

#define HIP_CHECK(expr) do { hipError_t _e = (expr); if (_e != hipSuccess) { \
    fprintf(stderr, "HIP error %s at %s:%d\n", hipGetErrorString(_e), __FILE__, __LINE__); \
    return; } } while (0)

extern "C" void kernel_launch(int rank, int nranks, ncclComm_t comm, hipStream_t stream,
                              void* const* d_in, const int* in_sizes, int n_in,
                              void* d_out, int out_size, const int* in_dtypes, int out_dtype,
                              void* const* d_out_all, void* const* d_ws_all, size_t ws_bytes) {
    (void)comm; (void)in_sizes; (void)n_in; (void)d_out; (void)in_dtypes; (void)out_dtype;
    if (nranks != NR || (unsigned)rank >= (unsigned)NR || ws_bytes < WS_NEED || out_size != NCOL * 4096) {
        fprintf(stderr, "unexpected problem geometry\n");
        return;
    }
    static bool attr_done[NR] = {false};
    if (!attr_done[rank]) {
        HIP_CHECK(hipFuncSetAttribute((const void*)fused_gemm_ar_q8, hipFuncAttributeMaxDynamicSharedMemorySize, LDS_BYTES));
        attr_done[rank] = true;
    }
    Params p;
    for (int r = 0; r < NR; ++r) { p.ws[r] = (char*)d_ws_all[r]; p.out[r] = (char*)d_out_all[r]; }
    p.x = (const char*)d_in[0];
    p.w = (const char*)d_in[1];
    p.rank = rank;
    hipLaunchKernelGGL(fused_gemm_ar_q8, dim3(NWG), dim3(NWAVE * 64), LDS_BYTES, stream, p);
    HIP_CHECK(hipGetLastError());
}
```
